```python
import numpy as np

kernel_name = "gpu_gemm_ar_m4096_k4096_n16384_w8a8_dequant"


ABSMAX_RTOL = 0.001

COMPUTE_DTYPE = "int8"
EPILOGUE = "dequant"
IN_QUANT = "int8"
OUT_QUANT = None
IN_DTYPES = {"x": "int8", "w_mat": "int8",
             "scale_x": "float32", "scale_w": "float32"}
WORLD_SIZE = 8

def setup_inputs():

    x = np.random.randint(-127, 128, size=(4096, 4096)).astype(np.int8)
    w_mat = np.random.randint(-127, 128, size=(4096, 16384)).astype(np.int8)
    scale_x = (0.02 * (1.0 + np.random.rand(1))).astype(np.float32)
    scale_w = (0.02 * (1.0 + np.random.rand(1))).astype(np.float32)
    return {"x": x, "w_mat": w_mat, "scale_x": scale_x, "scale_w": scale_w}

def reference(x, w_mat, scale_x, scale_w):
    acc = x.astype(np.float64) @ w_mat.astype(np.float64)
    y = (acc.astype(np.float32) * (scale_x * scale_w)).astype(np.float32)
    return y
```

```cpp
#include <hip/hip_runtime.h>
#include <rccl/rccl.h>
#include <cstdio>

#define HIP_CHECK(expr) do { hipError_t _e = (expr); if (_e != hipSuccess) { \
    fprintf(stderr, "HIP error %s at %s:%d\n", hipGetErrorString(_e), __FILE__, __LINE__); \
    return; } } while (0)

#define LAS __attribute__((address_space(3)))
typedef int i32x4 __attribute__((ext_vector_type(4)));
typedef unsigned u32x4 __attribute__((ext_vector_type(4)));
typedef float f32x4 __attribute__((ext_vector_type(4)));

namespace {
constexpr int M = 4096, N = 16384, K = 4096, NR = 8, KS = K / NR;
constexpr int HTB = 16384;
constexpr int NKS = K / 128;
constexpr size_t HB = (size_t)NKS * HTB;
constexpr size_t SLICE = 4 * (size_t)HTB;
constexpr int GRID = 256, NPULL = 24, NTHR = 512;
constexpr int NXCH = M / 128, NWCH = N / 128;
constexpr int NCHUNK = NXCH + NWCH;
constexpr int NBLK = M / 256 + N / 256;
constexpr int NPIECE = NBLK * 16;
constexpr unsigned READY_PER_CALL = 16 * 8;
constexpr int TILES_PER_Q = 128;

constexpr size_t OFF_CALL = 0, OFF_TMO = 128, OFF_STAGECTR = 256;
constexpr size_t OFF_TILECTR = 1024;
constexpr size_t OFF_STAGED = 4096;
constexpr size_t OFF_READY = 8192;
constexpr size_t OFF_DATA = 1ull << 20;
constexpr size_t XF_BYTES = (size_t)M * K, WF_BYTES = (size_t)N * K;
constexpr size_t PAR_BYTES = XF_BYTES + WF_BYTES;
constexpr size_t WS_NEED = OFF_DATA + 2 * PAR_BYTES;
constexpr unsigned long long DEADLINE_TICKS = 40000000ull;

struct Params {
    const signed char* x; const signed char* w; const float* sx; const float* sw; float* out;
    char* ws[NR];
    int rank;
};

__device__ __forceinline__ unsigned ld_sys(const unsigned* p) { return __hip_atomic_load(p, __ATOMIC_RELAXED, __HIP_MEMORY_SCOPE_SYSTEM); }
__device__ __forceinline__ unsigned ld_agt(const unsigned* p) { return __hip_atomic_load(p, __ATOMIC_RELAXED, __HIP_MEMORY_SCOPE_AGENT); }

template <bool SYS>
__device__ __forceinline__ bool spin_ge(const unsigned* flag, unsigned want, unsigned* tmo, unsigned call) {
    const unsigned long long t0 = __builtin_amdgcn_s_memrealtime();
    while ((int)((SYS ? ld_sys(flag) : ld_agt(flag)) - want) < 0) {
        if (ld_agt(tmo) == call || __builtin_amdgcn_s_memrealtime() - t0 > DEADLINE_TICKS) {
            __hip_atomic_store(tmo, call, __ATOMIC_RELAXED, __HIP_MEMORY_SCOPE_AGENT);
            return false;
        }
        __builtin_amdgcn_s_sleep(2);
    }
    return true;
}

__device__ __forceinline__ void tr4x4(unsigned a0, unsigned a1, unsigned a2, unsigned a3, unsigned& o0, unsigned& o1, unsigned& o2, unsigned& o3) {
    const unsigned t0 = __builtin_amdgcn_perm(a1, a0, 0x05010400u);
    const unsigned t1 = __builtin_amdgcn_perm(a1, a0, 0x07030602u);
    const unsigned t2 = __builtin_amdgcn_perm(a3, a2, 0x05010400u);
    const unsigned t3 = __builtin_amdgcn_perm(a3, a2, 0x07030602u);
    o0 = __builtin_amdgcn_perm(t2, t0, 0x05040100u);
    o1 = __builtin_amdgcn_perm(t2, t0, 0x07060302u);
    o2 = __builtin_amdgcn_perm(t3, t1, 0x05040100u);
    o3 = __builtin_amdgcn_perm(t3, t1, 0x07060302u);
}

__device__ __forceinline__ void st_sc1(void* p, u32x4 v) {
    asm volatile("global_store_dwordx4 %0, %1, off sc1\n\ts_nop 1" :: "v"(p), "v"(v) : "memory");
}

__global__ void __launch_bounds__(NTHR, 2) fused_k(Params P) {
    extern __shared__ __attribute__((aligned(16))) char smem_[];
    LAS char* lds = (LAS char*)smem_;
    LAS unsigned* lctl = (LAS unsigned*)(lds + 8 * HTB);
    const int tid = threadIdx.x, wid = __builtin_amdgcn_readfirstlane(tid >> 6), lane = tid & 63;
    const int bid = blockIdx.x, rank = P.rank;
    char* const myws = P.ws[rank];
    unsigned* const tmo = (unsigned*)(myws + OFF_TMO);

    if (tid == 0) {
        const unsigned t = __hip_atomic_fetch_add((unsigned*)(myws + OFF_CALL), 1u, __ATOMIC_RELAXED, __HIP_MEMORY_SCOPE_AGENT);
        lctl[0] = t / GRID + 1;
    }
    __syncthreads();
    const unsigned call = __builtin_amdgcn_readfirstlane(lctl[0]);
    const unsigned par = call & 1;
    const size_t dataoff = OFF_DATA + (size_t)par * PAR_BYTES;
    char* const XF = myws + dataoff; char* const WF = XF + XF_BYTES;

    if (bid < NCHUNK) {
        if (bid < NXCH) {
            const int c = bid;
            const signed char* src = P.x + (size_t)c * 128 * KS;
            char* dst = XF + (size_t)c * HB + (size_t)rank * SLICE;
            u32x4 v[8];
#pragma unroll
            for (int i = 0; i < 8; ++i) {
                const int o = i * NTHR + tid;
                const int l = o & 63, kf = (o >> 6) & 1, rg = (o >> 7) & 7, ks = o >> 10;
                v[i] = *(const u32x4*)(src + (size_t)(rg * 16 + (l & 15)) * KS + ks * 128 + kf * 64 + (l >> 4) * 16);
            }
#pragma unroll
            for (int i = 0; i < 8; ++i) *(u32x4*)(dst + (size_t)(i * NTHR + tid) * 16) = v[i];
        } else if (tid < 256) {
            const int c = bid - NXCH;
            const int nb = tid & 7, kg = tid >> 3;
            const signed char* src = P.w + (size_t)(kg * 16) * N + c * 128 + nb * 16;
            u32x4 r[16];
#pragma unroll
            for (int j = 0; j < 16; ++j) r[j] = *(const u32x4*)(src + (size_t)j * N);
            char* dst = WF + (size_t)c * HB + (size_t)rank * SLICE + (size_t)(kg >> 3) * HTB + (size_t)((nb * 2 + ((kg >> 2) & 1)) * 64 + (kg & 3) * 16) * 16;
#pragma unroll
            for (int d = 0; d < 4; ++d) {
                unsigned o[4][4];
#pragma unroll
                for (int g = 0; g < 4; ++g)
                    tr4x4(r[4 * g][d], r[4 * g + 1][d], r[4 * g + 2][d], r[4 * g + 3][d], o[0][g], o[1][g], o[2][g], o[3][g]);
#pragma unroll
                for (int i = 0; i < 4; ++i) *(u32x4*)(dst + (size_t)(4 * d + i) * 16) = (u32x4){o[i][0], o[i][1], o[i][2], o[i][3]};
            }
        }
        asm volatile("s_waitcnt vmcnt(0)" ::: "memory");
        __syncthreads();
        if (tid == 0) {
            __builtin_amdgcn_fence(__ATOMIC_RELEASE, "agent");
            asm volatile("s_waitcnt vmcnt(0)" ::: "memory");
            const unsigned t = __hip_atomic_fetch_add((unsigned*)(myws + OFF_STAGECTR), 1u, __ATOMIC_RELAXED, __HIP_MEMORY_SCOPE_AGENT);
            if (t + 1 == call * (unsigned)NCHUNK) {
                __builtin_amdgcn_fence(__ATOMIC_ACQUIRE, "agent");
                __builtin_amdgcn_fence(__ATOMIC_RELEASE, "");
                asm volatile("s_waitcnt vmcnt(0)" ::: "memory");
#pragma unroll
                for (int q = 1; q <= NR; ++q) {
                    const int p = (rank + q) & (NR - 1);
                    __hip_atomic_store((unsigned*)(P.ws[p] + OFF_STAGED + (size_t)rank * 128), call, __ATOMIC_RELAXED, __HIP_MEMORY_SCOPE_SYSTEM);
                }
            }
        }
    }
    if (bid == GRID - 1 && tid < 8)
        __hip_atomic_store((unsigned*)(myws + OFF_TILECTR + (size_t)((par ^ 1) * 8 + tid) * 128), 0u, __ATOMIC_RELAXED, __HIP_MEMORY_SCOPE_AGENT);

    if (bid < NPULL) {
        if (wid == 0) {
            if (lane < NR) spin_ge<true>((const unsigned*)(myws + OFF_STAGED + (size_t)lane * 128), call, tmo, call);
            __builtin_amdgcn_fence(__ATOMIC_ACQUIRE, "");
            asm volatile("s_waitcnt vmcnt(0)" ::: "memory");
        }
        __syncthreads();
        const bool dead = ld_agt(tmo) == call;
        if (!dead) {
            for (int g = bid; g < NPIECE; g += NPULL) {
                const int blk = g >> 4, sub = g & 15;
                const bool isx = blk < 32 && !(blk & 1);
                const int idx = blk < 32 ? (blk >> 1) : blk - 16;
                const size_t off = dataoff + (isx ? 0 : XF_BYTES) + (size_t)(idx * 2 + (sub >> 3)) * HB + (size_t)(sub & 7) * 8192 + (size_t)tid * 16;
                u32x4 v[7];
#pragma unroll
                for (int q = 1; q < NR; ++q) { const int p = (rank + q) & (NR - 1); v[q - 1] = *(const u32x4*)(P.ws[p] + off + (size_t)p * SLICE); }
#pragma unroll
                for (int q = 1; q < NR; ++q) { const int p = (rank + q) & (NR - 1); st_sc1(myws + off + (size_t)p * SLICE, v[q - 1]); }
                asm volatile("s_waitcnt vmcnt(0)" ::: "memory");
                if (lane == 0) __hip_atomic_fetch_add((unsigned*)(myws + OFF_READY + (size_t)(isx ? idx : 16 + idx) * 128), 1u, __ATOMIC_RELAXED, __HIP_MEMORY_SCOPE_AGENT);
            }
        }
        __syncthreads();
    }

    const int wr = wid >> 2, wc = wid & 3, fr = lane & 15, fq = lane >> 4;
    const unsigned voff = (unsigned)(wid * 1024 + lane * 16);
    const unsigned ldsw = (unsigned)wid * 1024u;
    const int aoff = wr * 8192 + lane * 16, boff = wc * 4096 + lane * 16;
    const int xcd = bid & 7;
    const float scale0 = P.sx[0] * P.sw[0];
    unsigned* const tilectr = (unsigned*)(myws + OFF_TILECTR + (size_t)(par * 8 + xcd) * 128);
    const unsigned need = call * READY_PER_CALL;
#define SA(b, h) (((b) * 2 + (h)) * HTB)
#define SB(b, h) ((4 + (b) * 2 + (h)) * HTB)
#define STAGE(bufoff, gbase) do { _Pragma("unroll") for (int _i = 0; _i < 2; ++_i) \
        __builtin_amdgcn_global_load_lds((const unsigned*)((const char*)(gbase) + voff + _i * 8192), (LAS unsigned*)(lds + (bufoff) + ldsw + _i * 8192), 16, 0, 0); } while (0)
#define LDA(dst, b, h) do { _Pragma("unroll") for (int m = 0; m < 4; ++m) _Pragma("unroll") for (int k = 0; k < 2; ++k) dst[m][k] = *(const LAS i32x4*)(lds + SA(b, h) + aoff + m * 2048 + k * 1024); } while (0)
#define LDB(dst, b, h) do { _Pragma("unroll") for (int n = 0; n < 2; ++n) _Pragma("unroll") for (int k = 0; k < 2; ++k) dst[n][k] = *(const LAS i32x4*)(lds + SB(b, h) + boff + n * 2048 + k * 1024); } while (0)
#define MMA(ai, bj, At, Bt) do { __builtin_amdgcn_s_setprio(1); _Pragma("unroll") for (int m = 0; m < 4; ++m) _Pragma("unroll") for (int n = 0; n < 2; ++n) _Pragma("unroll") for (int k = 0; k < 2; ++k) \
        acc[ai][bj][m][n] = __builtin_amdgcn_mfma_i32_16x16x64_i8(Bt[n][k], At[m][k], acc[ai][bj][m][n], 0, 0, 0); __builtin_amdgcn_s_setprio(0); } while (0)
#define WAIT_V(n) asm volatile("s_waitcnt vmcnt(" #n ")" ::: "memory")
#define WAIT_L(n) asm volatile("s_waitcnt lgkmcnt(" #n ")" ::: "memory")
#define BAR __builtin_amdgcn_s_barrier()
#define SCHED __builtin_amdgcn_sched_barrier(0)

    for (;;) {
        if (tid == 0) {
            const unsigned e = __hip_atomic_fetch_add(tilectr, 1u, __ATOMIC_RELAXED, __HIP_MEMORY_SCOPE_AGENT);
            unsigned code = 0xffffffffu;
            if (e < (unsigned)TILES_PER_Q) {
                int ii, jj;
                if (e >= 32) { ii = e & 3; jj = e >> 2; }
                else {
                    int t = 0; unsigned r = e; while (r >= (unsigned)(2 * (2 * t + 1))) { r -= 2 * (2 * t + 1); ++t; }
                    if (r < (unsigned)(2 * t)) { ii = t; jj = r; }
                    else { r -= 2 * t; if (r <= (unsigned)t) { ii = r; jj = 2 * t; } else { ii = r - t - 1; jj = 2 * t + 1; } }
                }
                const int pm_ = (xcd & 3) + 4 * ii, pn_ = (xcd >> 2) + 2 * jj;
                bool ok = spin_ge<false>((const unsigned*)(myws + OFF_READY + (size_t)pm_ * 128), need, tmo, call);
                ok = spin_ge<false>((const unsigned*)(myws + OFF_READY + (size_t)(16 + pn_) * 128), need, tmo, call) && ok;
                if (ld_agt(tmo) == call) ok = false;
                __builtin_amdgcn_fence(__ATOMIC_ACQUIRE, "agent");
                asm volatile("s_waitcnt vmcnt(0)" ::: "memory");
                code = (unsigned)pm_ | ((unsigned)pn_ << 8) | (ok ? 0u : 0x10000u);
            }
            lctl[1] = code;
        }
        __syncthreads();
        const unsigned code = __builtin_amdgcn_readfirstlane(lctl[1]);
        __syncthreads();
        if (code == 0xffffffffu) break;
        const int pm = code & 0xff, pn = (code >> 8) & 0xff;
        const float scale = (code & 0x10000u) ? __builtin_nanf("") : scale0;

        i32x4 acc[2][2][4][2];
#pragma unroll
        for (int a = 0; a < 2; ++a)
#pragma unroll
            for (int b = 0; b < 2; ++b)
#pragma unroll
                for (int m = 0; m < 4; ++m)
#pragma unroll
                    for (int n = 0; n < 2; ++n) acc[a][b][m][n] = (i32x4){0, 0, 0, 0};
        i32x4 At[4][2], B0[2][2], B1[2][2];
        const char* cA = XF + (size_t)pm * 2 * HB; const char* cB = WF + (size_t)pn * 2 * HB;
        STAGE(SB(0, 0), cB); STAGE(SA(0, 0), cA); STAGE(SB(0, 1), cB + HB); STAGE(SA(0, 1), cA + HB);
        if (wr == 1) BAR;
        WAIT_V(4); BAR;
        STAGE(SB(1, 0), cB + HTB); STAGE(SA(1, 0), cA + HTB); STAGE(SB(1, 1), cB + HB + HTB);
        WAIT_V(6); BAR;
        for (int t = 0; t < NKS; t += 2) {
            const bool last = (t == NKS - 2);
            const char* a1 = cA + (size_t)(t + 1) * HTB;
            const char* a2 = last ? cA : cA + (size_t)(t + 2) * HTB; const char* b2 = last ? cB : cB + (size_t)(t + 2) * HTB;
            const char* a3 = a2 + HTB; const char* b3 = b2 + HTB;
            LDB(B0, 0, 0); SCHED; LDA(At, 0, 0); STAGE(SA(1, 1), a1 + HB);
            WAIT_L(8); WAIT_V(10); BAR; WAIT_L(0); MMA(0, 0, At, B0); BAR; SCHED;
            LDB(B1, 0, 1); STAGE(SB(0, 0), b2);
            WAIT_V(10); BAR; WAIT_L(0); MMA(0, 1, At, B1); BAR;
            LDA(At, 0, 1); STAGE(SA(0, 0), a2);
            BAR; WAIT_L(0); MMA(1, 0, At, B0); BAR; SCHED;
            STAGE(SB(0, 1), b2 + HB);
            WAIT_V(10); BAR; MMA(1, 1, At, B1); BAR;
            LDB(B0, 1, 0); SCHED; LDA(At, 1, 0); STAGE(SA(0, 1), a2 + HB);
            WAIT_L(8); WAIT_V(10); BAR; WAIT_L(0); MMA(0, 0, At, B0); BAR; SCHED;
            LDB(B1, 1, 1); STAGE(SB(1, 0), b3);
            WAIT_V(10); BAR; WAIT_L(0); MMA(0, 1, At, B1); BAR;
            LDA(At, 1, 1); STAGE(SA(1, 0), a3);
            BAR; WAIT_L(0); MMA(1, 0, At, B0); BAR; SCHED;
            STAGE(SB(1, 1), b3 + HB);
            WAIT_V(10); BAR; MMA(1, 1, At, B1); BAR;
        }
        {
            const int row0 = pm * 256 + wr * 64 + fr, col0 = pn * 256 + wc * 32 + 4 * fq;
#pragma unroll
            for (int ai = 0; ai < 2; ++ai)
#pragma unroll
                for (int m = 0; m < 4; ++m) { float* rowp = P.out + (size_t)(row0 + ai * 128 + m * 16) * N + col0;
#pragma unroll
                    for (int bj = 0; bj < 2; ++bj)
#pragma unroll
                        for (int n = 0; n < 2; ++n) { const i32x4 v = acc[ai][bj][m][n];
                            *(f32x4*)(rowp + bj * 128 + n * 16) = (f32x4){(float)v[0] * scale, (float)v[1] * scale, (float)v[2] * scale, (float)v[3] * scale}; } }
        }
        WAIT_V(0);
        if (wr == 0) BAR;
        BAR;
    }
}
}

extern "C" void kernel_launch(int rank, int nranks, ncclComm_t comm,
                              hipStream_t stream,
                              void* const* d_in, const int* in_sizes, int n_in,
                              void* d_out, int out_size,
                              const int* in_dtypes, int out_dtype,
                              void* const* d_out_all,
                              void* const* d_ws_all, size_t ws_bytes) {
    (void)comm; (void)in_sizes; (void)n_in; (void)out_size; (void)in_dtypes; (void)out_dtype; (void)d_out_all;
    if (nranks != NR || ws_bytes < WS_NEED) { fprintf(stderr, "unexpected configuration\n"); return; }
    static bool attr_set = false;
    constexpr int LDS_BYTES = 8 * HTB + 64;
    if (!attr_set) {
        HIP_CHECK(hipFuncSetAttribute((const void*)fused_k, hipFuncAttributeMaxDynamicSharedMemorySize, LDS_BYTES));
        attr_set = true;
    }
    Params P;
    P.x = (const signed char*)d_in[0]; P.w = (const signed char*)d_in[1];
    P.sx = (const float*)d_in[2]; P.sw = (const float*)d_in[3];
    P.out = (float*)d_out; P.rank = rank;
    for (int p = 0; p < NR; ++p) P.ws[p] = (char*)d_ws_all[p];
    fused_k<<<GRID, NTHR, LDS_BYTES, stream>>>(P);
    HIP_CHECK(hipGetLastError());
}
```

```cpp
#include <hip/hip_runtime.h>
#include <rccl/rccl.h>
#include <cstdio>

#define HIP_CHECK(expr) do { hipError_t _e = (expr); if (_e != hipSuccess) { \
    fprintf(stderr, "HIP error %s at %s:%d\n", hipGetErrorString(_e), __FILE__, __LINE__); \
    return; } } while (0)

#define LAS __attribute__((address_space(3)))
typedef int i32x4 __attribute__((ext_vector_type(4)));
typedef unsigned u32x4 __attribute__((ext_vector_type(4)));
typedef float f32x4 __attribute__((ext_vector_type(4)));

namespace {
constexpr int M = 4096, N = 16384, K = 4096, NR = 8, KS = K / NR;
constexpr int HTB = 16384;
constexpr int NKS = K / 128;
constexpr size_t HB = (size_t)NKS * HTB;
constexpr size_t SLICE = 4 * (size_t)HTB;
constexpr int GRID = 256, NPULL = 24, NTHR = 512;
constexpr int NXCH = M / 128, NWCH = N / 128;
constexpr int NCHUNK = NXCH + NWCH;
constexpr int NBLK = M / 256 + N / 256;
constexpr int NPIECE = NBLK * 16;
constexpr unsigned READY_PER_CALL = 16 * 8;
constexpr int TILES_PER_Q = 128;
#define NSPLIT0_ 18
#define NSPLIT1_ 16
constexpr int NSPLIT0 = NSPLIT0_, NSPLIT1 = NSPLIT1_;
#define KLEAD 2u
constexpr int NJOBS_Q = TILES_PER_Q + NSPLIT0 + NSPLIT1;

constexpr size_t OFF_CALL = 0, OFF_TMO = 128, OFF_STAGECTR = 256;
constexpr size_t OFF_TILECTR = 1024;
constexpr size_t OFF_PHASE = 3072;
constexpr size_t OFF_STAGED = 4096;
constexpr size_t OFF_READY = 8192;
constexpr size_t OFF_DBG = 65536;
constexpr size_t OFF_DATA = 1ull << 20;
constexpr size_t XF_BYTES = (size_t)M * K, WF_BYTES = (size_t)N * K;
constexpr size_t PAR_BYTES = XF_BYTES + WF_BYTES;
constexpr size_t WS_NEED = OFF_DATA + 2 * PAR_BYTES;
constexpr unsigned long long DEADLINE_TICKS = 40000000ull;

struct Params {
    const signed char* x; const signed char* w; const float* sx; const float* sw; float* out;
    char* ws[NR];
    int rank;
};

__device__ __forceinline__ unsigned ld_sys(const unsigned* p) { return __hip_atomic_load(p, __ATOMIC_RELAXED, __HIP_MEMORY_SCOPE_SYSTEM); }
__device__ __forceinline__ unsigned ld_agt(const unsigned* p) { return __hip_atomic_load(p, __ATOMIC_RELAXED, __HIP_MEMORY_SCOPE_AGENT); }

template <bool SYS>
__device__ __forceinline__ bool spin_ge(const unsigned* flag, unsigned want, unsigned* tmo, unsigned call) {
    const unsigned long long t0 = __builtin_amdgcn_s_memrealtime();
    while ((int)((SYS ? ld_sys(flag) : ld_agt(flag)) - want) < 0) {
        if (ld_agt(tmo) == call || __builtin_amdgcn_s_memrealtime() - t0 > DEADLINE_TICKS) {
            __hip_atomic_store(tmo, call, __ATOMIC_RELAXED, __HIP_MEMORY_SCOPE_AGENT);
            return false;
        }
        __builtin_amdgcn_s_sleep(2);
    }
    return true;
}

__device__ __forceinline__ void tr4x4(unsigned a0, unsigned a1, unsigned a2, unsigned a3, unsigned& o0, unsigned& o1, unsigned& o2, unsigned& o3) {
    const unsigned t0 = __builtin_amdgcn_perm(a1, a0, 0x05010400u);
    const unsigned t1 = __builtin_amdgcn_perm(a1, a0, 0x07030602u);
    const unsigned t2 = __builtin_amdgcn_perm(a3, a2, 0x05010400u);
    const unsigned t3 = __builtin_amdgcn_perm(a3, a2, 0x07030602u);
    o0 = __builtin_amdgcn_perm(t2, t0, 0x05040100u);
    o1 = __builtin_amdgcn_perm(t2, t0, 0x07060302u);
    o2 = __builtin_amdgcn_perm(t3, t1, 0x05040100u);
    o3 = __builtin_amdgcn_perm(t3, t1, 0x07060302u);
}

__device__ __forceinline__ void st_sc1(void* p, u32x4 v) {
    asm volatile("global_store_dwordx4 %0, %1, off sc1\n\ts_nop 1" :: "v"(p), "v"(v) : "memory");
}

#define TS() do {} while (0)
__global__ void __launch_bounds__(NTHR, 2) fused_k(Params P) {
    TS();
    extern __shared__ __attribute__((aligned(16))) char smem_[];
    LAS char* lds = (LAS char*)smem_;
    LAS unsigned* lctl = (LAS unsigned*)(lds + 9 * HTB);
    const int tid = threadIdx.x, wid = __builtin_amdgcn_readfirstlane(tid >> 6), lane = tid & 63;
    const int bid = blockIdx.x, rank = P.rank;
    char* const myws = P.ws[rank];
    unsigned* const tmo = (unsigned*)(myws + OFF_TMO);

    u32x4 sv[16];
    const bool stx = bid < NXCH, stw = bid >= NXCH && bid < NCHUNK && tid < 256;
    if (stx) {
        const signed char* src = P.x + (size_t)bid * 128 * KS;
#pragma unroll
        for (int i = 0; i < 8; ++i) {
            const int o = i * NTHR + tid;
            const int l = o & 63, kf = (o >> 6) & 1, rg = (o >> 7) & 7, ks = o >> 10;
            sv[i] = *(const u32x4*)(src + (size_t)(rg * 16 + (l & 15)) * KS + ks * 128 + kf * 64 + (l >> 4) * 16);
        }
    } else if (stw) {
        const int cg = tid & 15, kg = (tid >> 4) + ((bid - NXCH) & 1) * 16;
        const signed char* src = P.w + (size_t)(kg * 16) * N + ((bid - NXCH) >> 1) * 256 + cg * 16;
#pragma unroll
        for (int j = 0; j < 16; ++j) sv[j] = *(const u32x4*)(src + (size_t)j * N);
    }
    if (tid == 0) {
        const unsigned t = __hip_atomic_fetch_add((unsigned*)(myws + OFF_CALL), 1u, __ATOMIC_RELAXED, __HIP_MEMORY_SCOPE_AGENT);
        lctl[0] = t / GRID + 1;
    }
    __syncthreads();
    const unsigned call = __builtin_amdgcn_readfirstlane(lctl[0]);
    const unsigned par = call & 1;
    const size_t dataoff = OFF_DATA + (size_t)par * PAR_BYTES;
    char* const XF = myws + dataoff; char* const WF = XF + XF_BYTES;
    const __amdgpu_buffer_rsrc_t drs = __builtin_amdgcn_make_buffer_rsrc(XF, 0, (unsigned)PAR_BYTES, 0x00020000);

    if (bid < NCHUNK) {
        if (stx) {
            const unsigned dst = (unsigned)((size_t)bid * HB + (size_t)rank * SLICE);
#pragma unroll
            for (int i = 0; i < 8; ++i) __builtin_amdgcn_raw_buffer_store_b128(sv[i], drs, dst + (unsigned)(i * NTHR + tid) * 16u, 0, 17);
        } else {
            if (stw) {
                const int cg = tid & 15, kl = tid >> 4;
                const int dst = (kl >> 3) * HTB + (((cg >> 2) * 2) * 2 + ((kl >> 2) & 1)) * 1024 + ((kl & 3) * 16 + (cg & 3) * 4) * 16;
#pragma unroll
                for (int d = 0; d < 4; ++d) {
                    unsigned o[4][4];
#pragma unroll
                    for (int g = 0; g < 4; ++g)
                        tr4x4(sv[4 * g][d], sv[4 * g + 1][d], sv[4 * g + 2][d], sv[4 * g + 3][d], o[0][g], o[1][g], o[2][g], o[3][g]);
#pragma unroll
                    for (int i = 0; i < 4; ++i)
                        *(LAS u32x4*)(lds + dst + (i >> 1) * 2 * HTB + (i & 1) * 2048 + d * 16) = (u32x4){o[i][0], o[i][1], o[i][2], o[i][3]};
                }
            }
            __syncthreads();
            const unsigned dst = (unsigned)(XF_BYTES + (size_t)((bid - NXCH) >> 1) * 2 * HB + (size_t)rank * SLICE + (size_t)((bid - NXCH) & 1) * 2 * HTB);
#pragma unroll
            for (int i = 0; i < 8; ++i)
                __builtin_amdgcn_raw_buffer_store_b128(*(const LAS u32x4*)(lds + (i * NTHR + tid) * 16), drs, dst + (unsigned)(i >> 2) * (unsigned)HB + (unsigned)((i & 3) * NTHR + tid) * 16u, 0, 17);
        }
        asm volatile("s_waitcnt vmcnt(0)" ::: "memory");
        __syncthreads();
        if (tid == 0) {
            const unsigned t = __hip_atomic_fetch_add((unsigned*)(myws + OFF_STAGECTR), 1u, __ATOMIC_RELAXED, __HIP_MEMORY_SCOPE_AGENT);
            if (t + 1 == call * (unsigned)NCHUNK) {
#pragma unroll
                for (int q = 1; q <= NR; ++q) {
                    const int p = (rank + q) & (NR - 1);
                    __hip_atomic_store((unsigned*)(P.ws[p] + OFF_STAGED + (size_t)rank * 128), call, __ATOMIC_RELAXED, __HIP_MEMORY_SCOPE_SYSTEM);
                }
            }
        }
    }
    TS();
    if (bid == GRID - 1 && tid < 8)
        __hip_atomic_store((unsigned*)(myws + OFF_TILECTR + (size_t)((par ^ 1) * 8 + tid) * 128), 0u, __ATOMIC_RELAXED, __HIP_MEMORY_SCOPE_AGENT);

    if (bid < NPULL) {
        if (wid == 0) {
            if (lane < NR) spin_ge<true>((const unsigned*)(myws + OFF_STAGED + (size_t)lane * 128), call, tmo, call);
            __builtin_amdgcn_fence(__ATOMIC_ACQUIRE, "");
            asm volatile("s_waitcnt vmcnt(0)" ::: "memory");
        }
        __syncthreads();
        const bool dead = ld_agt(tmo) == call;
        TS();
        if (!dead) {
            auto piece_off = [&](int g, unsigned& rdy) -> unsigned {
                const int blk = g >> 4, sub = g & 15;
                const bool isx = blk < 32 && !(blk & 1);
                const int idx = blk < 32 ? (blk >> 1) : blk - 16;
                rdy = (unsigned)(isx ? idx : 16 + idx);
                return (unsigned)((isx ? 0 : XF_BYTES) + (size_t)(idx * 2 + (sub >> 3)) * HB + (size_t)(sub & 7) * 8192 + (size_t)tid * 16);
            };
            u32x4 v[7];
            unsigned rdy, off = piece_off(bid, rdy);
#pragma unroll
            for (int q = 1; q < NR; ++q) { const int p = (rank + q) & (NR - 1); v[q - 1] = *(const u32x4*)(P.ws[p] + dataoff + off + (size_t)p * SLICE); }
            for (int g = bid; g < NPIECE; g += NPULL) {
#pragma unroll
                for (int q = 1; q < NR; ++q) { const int p = (rank + q) & (NR - 1); __builtin_amdgcn_raw_buffer_store_b128(v[q - 1], drs, off + (unsigned)(p * SLICE), 0, 16); }
                const unsigned rdy_cur = rdy;
                const int gn = g + NPULL < NPIECE ? g + NPULL : g;
                off = piece_off(gn, rdy);
#pragma unroll
                for (int q = 1; q < NR; ++q) { const int p = (rank + q) & (NR - 1); v[q - 1] = *(const u32x4*)(P.ws[p] + dataoff + off + (size_t)p * SLICE); }
                asm volatile("s_waitcnt vmcnt(7)" ::: "memory");
                if (lane == 0) __hip_atomic_fetch_add((unsigned*)(myws + OFF_READY + (size_t)rdy_cur * 128), 1u, __ATOMIC_RELAXED, __HIP_MEMORY_SCOPE_AGENT);
            }
            asm volatile("s_waitcnt vmcnt(0)" ::: "memory");
        }
        __syncthreads();
        TS();
    }

    const int wr = wid >> 2, wc = wid & 3, fr = lane & 15, fq = lane >> 4;
    const unsigned voff = (unsigned)(wid * 1024 + lane * 16);
    const unsigned ldsw = (unsigned)wid * 1024u;
    const int aoff = wr * 8192 + lane * 16, boff = wc * 4096 + lane * 16;
    const int xcd = bid & 7;
    const float scale0 = P.sx[0] * P.sw[0];
    unsigned* const tilectr = (unsigned*)(myws + OFF_TILECTR + (size_t)(par * 8 + xcd) * 128);
    const unsigned need = call * READY_PER_CALL;
    unsigned* const phase = (unsigned*)(myws + OFF_PHASE + (size_t)xcd * 128);
    int k0 = 0;
#define SA(b, h) (((b) * 2 + (h)) * HTB)
#define SB(b, h) ((4 + (b) * 2 + (h)) * HTB)
#define STAGE(bufoff, gbase) do { _Pragma("unroll") for (int _i = 0; _i < 2; ++_i) \
        __builtin_amdgcn_global_load_lds((const unsigned*)((const char*)(gbase) + voff + _i * 8192), (LAS unsigned*)(lds + (bufoff) + ldsw + _i * 8192), 16, 0, 0); } while (0)
#define LDA(dst, b, h) do { _Pragma("unroll") for (int m = 0; m < 4; ++m) _Pragma("unroll") for (int k = 0; k < 2; ++k) dst[m][k] = *(const LAS i32x4*)(lds + SA(b, h) + aoff + m * 2048 + k * 1024); } while (0)
#define LDB(dst, b, h) do { _Pragma("unroll") for (int n = 0; n < 2; ++n) _Pragma("unroll") for (int k = 0; k < 2; ++k) dst[n][k] = *(const LAS i32x4*)(lds + SB(b, h) + boff + n * 2048 + k * 1024); } while (0)
#define MMA(ai, bj, At, Bt) do { __builtin_amdgcn_s_setprio(1); _Pragma("unroll") for (int m = 0; m < 4; ++m) _Pragma("unroll") for (int n = 0; n < 2; ++n) _Pragma("unroll") for (int k = 0; k < 2; ++k) \
        acc[ai][bj][m][n] = __builtin_amdgcn_mfma_i32_16x16x64_i8(At[m][k], Bt[n][k], acc[ai][bj][m][n], 0, 0, 0); __builtin_amdgcn_s_setprio(0); } while (0)
#define WAIT_V(n) asm volatile("s_waitcnt vmcnt(" #n ")" ::: "memory")
#define WAIT_L(n) asm volatile("s_waitcnt lgkmcnt(" #n ")" ::: "memory")
#define BAR __builtin_amdgcn_s_barrier()
#define SCHED __builtin_amdgcn_sched_barrier(0)

    constexpr unsigned NOJOB = 0xffffffffu;
    auto decode = [&](unsigned e) -> unsigned {
        if (e >= (unsigned)NJOBS_Q) return NOJOB;
        unsigned u, mode;
        if (e < 2u * NSPLIT0) { u = e >> 1; mode = 1 + (e & 1); }
        else if (e < (unsigned)(TILES_PER_Q + NSPLIT0 - NSPLIT1)) { u = e - NSPLIT0; mode = 0; }
        else { const unsigned r = e - (unsigned)(TILES_PER_Q + NSPLIT0 - NSPLIT1); u = TILES_PER_Q - NSPLIT1 + (r >> 1); mode = 1 + (r & 1); }
        int ii, jj;
        if (u >= 32) { ii = u & 3; jj = u >> 2; }
        else {
            int t = 0; unsigned r = u; while (r >= (unsigned)(2 * (2 * t + 1))) { r -= 2 * (2 * t + 1); ++t; }
            if (r < (unsigned)(2 * t)) { ii = t; jj = r; }
            else { r -= 2 * t; if (r <= (unsigned)t) { ii = r; jj = 2 * t; } else { ii = r - t - 1; jj = 2 * t + 1; } }
        }
        return (unsigned)((xcd & 3) + 4 * ii) | ((unsigned)((xcd >> 2) + 2 * jj) << 8) | (mode << 16);
    };

    if (tid == 0) {
        const unsigned e = __hip_atomic_fetch_add(tilectr, 1u, __ATOMIC_RELAXED, __HIP_MEMORY_SCOPE_AGENT);
        lctl[1] = decode(e);
    }
    __syncthreads();
    unsigned cur = __builtin_amdgcn_readfirstlane(lctl[1]);
    bool prefetched = false, bad = false;
    int hset = 0;
    i32x4 acc[2][2][4][2];
    i32x4 At[4][2], B0[2][2], B1[2][2];

    while (cur != NOJOB) {
        const int pm = cur & 0xff, pn = (cur >> 8) & 0xff, mode = cur >> 16;
        const char* cA = XF + (size_t)(pm * 2 + (mode == 2)) * HB;
        const char* cB = WF + (size_t)pn * 2 * HB;
        if (!prefetched) {
            if (tid == 0) {
                bool ok = spin_ge<false>((const unsigned*)(myws + OFF_READY + (size_t)pm * 128), need, tmo, call);
                ok = spin_ge<false>((const unsigned*)(myws + OFF_READY + (size_t)(16 + pn) * 128), need, tmo, call) && ok;
                if (ld_agt(tmo) == call) ok = false;
                __builtin_amdgcn_fence(__ATOMIC_ACQUIRE, "agent");
                asm volatile("s_waitcnt vmcnt(0)" ::: "memory");
                lctl[3] = ok ? 0u : 1u;
                lctl[5] = (ld_agt(phase) + KLEAD) & (unsigned)(NKS - 2);
            }
            __syncthreads();
            bad = bad || (__builtin_amdgcn_readfirstlane(lctl[3]) != 0u);
            k0 = __builtin_amdgcn_readfirstlane(lctl[5]);
            TS();
            if (mode == 0) {
                const char* pA = cA + (size_t)k0 * HTB; const char* pB = cB + (size_t)k0 * HTB;
                STAGE(SB(0, 0), pB); STAGE(SA(0, 0), pA); STAGE(SB(0, 1), pB + HB); STAGE(SA(0, 1), pA + HB);
                if (wr == 1) BAR;
                WAIT_V(4); BAR;
                STAGE(SB(1, 0), pB + HTB); STAGE(SA(1, 0), pA + HTB); STAGE(SB(1, 1), pB + HB + HTB);
                WAIT_V(6); BAR;
            } else {
                const char* pA = cA + (size_t)k0 * HTB; const char* pB = cB + (size_t)k0 * HTB;
                STAGE(0, pA); STAGE(HTB, pB); STAGE(2 * HTB, pB + HB);
                STAGE(3 * HTB, pA + HTB); STAGE(4 * HTB, pB + HTB); STAGE(5 * HTB, pB + HB + HTB);
                if (wr == 1) BAR;
                WAIT_V(8); BAR; BAR;
                hset = 0;
            }
        }
#pragma unroll
        for (int a = 0; a < 2; ++a)
#pragma unroll
            for (int b = 0; b < 2; ++b)
#pragma unroll
                for (int m = 0; m < 4; ++m)
#pragma unroll
                    for (int n = 0; n < 2; ++n) acc[a][b][m][n] = (i32x4){0, 0, 0, 0};
        unsigned nxt = NOJOB, tk = 0;
        const char* nA = cA; const char* nB = cB;
        bool seam = false;
#define TICKET() do { if (wid == 0 && lane == 0) { const unsigned one = 1u; asm volatile("global_atomic_add %0, %1, %2, off sc0" : "+v"(tk) : "v"(tilectr), "v"(one) : "memory"); } } while (0)
#define PROBE() do { if (wid == 0) { asm volatile("" : "+v"(tk)); const unsigned c_ = decode(__builtin_amdgcn_readfirstlane(tk)); \
            if (lane == 0) lctl[4] = c_; \
            const unsigned* probe = (const unsigned*)(myws + (lane == 0 ? OFF_READY + (size_t)(c_ & 15) * 128 : lane == 1 ? OFF_READY + (size_t)(16 + ((c_ >> 8) & 63)) * 128 : OFF_TMO)); \
            if (lane < 3) __builtin_amdgcn_global_load_lds(probe, (LAS unsigned*)(lctl + 8), 4, 0, 16); } } while (0)
#define SEAM() do { nxt = __builtin_amdgcn_readfirstlane(lctl[4]); const unsigned r0 = lctl[8], r1 = lctl[9], r2 = lctl[10]; \
            const bool cand = nxt != NOJOB && (((nxt >> 16) == 0) == (mode == 0)); \
            seam = __builtin_amdgcn_readfirstlane((int)(cand && (int)(r0 - need) >= 0 && (int)(r1 - need) >= 0 && r2 != call)) != 0; \
            if (seam) { nA = XF + (size_t)((nxt & 15) * 2 + ((nxt >> 16) == 2)) * HB; nB = WF + (size_t)((nxt >> 8) & 63) * 2 * HB; } } while (0)
        if (mode == 0) {
            for (int t = 0; t < NKS; t += 2) {
                const bool last = (t == NKS - 2);
                if (last) SEAM();
                const int kc = (k0 + t) & (NKS - 1), kn = (k0 + t + 2) & (NKS - 1);
                const char* a1 = cA + (size_t)(kc + 1) * HTB;
                const char* a2 = (last ? nA : cA) + (size_t)kn * HTB; const char* b2 = (last ? nB : cB) + (size_t)kn * HTB;
                const char* a3 = a2 + HTB; const char* b3 = b2 + HTB;
                if (tid == 0) __hip_atomic_store(phase, (unsigned)kc, __ATOMIC_RELAXED, __HIP_MEMORY_SCOPE_AGENT);
                if (t == NKS - 6) TICKET();
                if (t == NKS - 4) PROBE();
                LDB(B0, 0, 0); SCHED; LDA(At, 0, 0); STAGE(SA(1, 1), a1 + HB);
                WAIT_L(8); WAIT_V(10); BAR; WAIT_L(0); MMA(0, 0, At, B0); BAR; SCHED;
                LDB(B1, 0, 1); STAGE(SB(0, 0), b2);
                WAIT_V(10); BAR; WAIT_L(0); MMA(0, 1, At, B1); BAR;
                LDA(At, 0, 1); STAGE(SA(0, 0), a2);
                BAR; WAIT_L(0); MMA(1, 0, At, B0); BAR; SCHED;
                STAGE(SB(0, 1), b2 + HB);
                WAIT_V(10); BAR; MMA(1, 1, At, B1); BAR;
                LDB(B0, 1, 0); SCHED; LDA(At, 1, 0); STAGE(SA(0, 1), a2 + HB);
                WAIT_L(8); WAIT_V(10); BAR; WAIT_L(0); MMA(0, 0, At, B0); BAR; SCHED;
                LDB(B1, 1, 1); STAGE(SB(1, 0), b3);
                WAIT_V(10); BAR; WAIT_L(0); MMA(0, 1, At, B1); BAR;
                LDA(At, 1, 1); STAGE(SA(1, 0), a3);
                BAR; WAIT_L(0); MMA(1, 0, At, B0); BAR; SCHED;
                STAGE(SB(1, 1), b3 + HB);
                WAIT_V(10); BAR; MMA(1, 1, At, B1); BAR;
            }
        } else {
            for (int t = 0; t < NKS; ++t) {
                if (t == NKS - 2) SEAM();
                const int kn = (k0 + t + 2) & (NKS - 1);
                const char* a2 = (t + 2 < NKS ? cA : nA) + (size_t)kn * HTB;
                const char* b2 = (t + 2 < NKS ? cB : nB) + (size_t)kn * HTB;
                const int hst = hset == 0 ? 6 * HTB : hset - 3 * HTB;
                if (t == NKS - 10) TICKET();
                if (t == NKS - 6) PROBE();
#pragma unroll
                for (int n = 0; n < 2; ++n)
#pragma unroll
                    for (int k = 0; k < 2; ++k) B0[n][k] = *(const LAS i32x4*)(lds + hset + HTB + boff + n * 2048 + k * 1024);
#pragma unroll
                for (int m = 0; m < 4; ++m)
#pragma unroll
                    for (int k = 0; k < 2; ++k) At[m][k] = *(const LAS i32x4*)(lds + hset + aoff + m * 2048 + k * 1024);
                STAGE(hst, a2); STAGE(hst + HTB, b2);
                WAIT_V(10); BAR; WAIT_L(0); MMA(0, 0, At, B0); BAR; SCHED;
#pragma unroll
                for (int n = 0; n < 2; ++n)
#pragma unroll
                    for (int k = 0; k < 2; ++k) B1[n][k] = *(const LAS i32x4*)(lds + hset + 2 * HTB + boff + n * 2048 + k * 1024);
                STAGE(hst + 2 * HTB, b2 + HB);
                WAIT_V(8); BAR; WAIT_L(0); MMA(0, 1, At, B1); BAR; SCHED;
                hset = hset == 6 * HTB ? 0 : hset + 3 * HTB;
            }
        }
        {
            const float scale = bad ? __builtin_nanf("") : scale0;
            float* op = P.out + (size_t)(pm * 256 + (mode == 2 ? 128 : 0) + wr * 64 + 4 * fq) * N + pn * 256 + wc * 64 + 4 * fr;
#pragma unroll
            for (int ai = 0; ai < 2; ++ai) {
                if (ai == 1 && mode != 0) break;
#pragma unroll
                for (int m = 0; m < 4; ++m)
#pragma unroll
                    for (int j = 0; j < 4; ++j)
                        *(f32x4*)(op + (size_t)(ai * 128 + m * 16 + j) * N) = (f32x4){(float)acc[ai][0][m][0][j] * scale, (float)acc[ai][0][m][1][j] * scale,
                                                                                     (float)acc[ai][1][m][0][j] * scale, (float)acc[ai][1][m][1][j] * scale};
            }
        }
        TS();
        cur = nxt;
        prefetched = seam;
        if (!seam) {
            WAIT_V(0);
            if (wr == 0) BAR;
            BAR;
        }
    }
}
}

extern "C" void kernel_launch(int rank, int nranks, ncclComm_t comm,
                              hipStream_t stream,
                              void* const* d_in, const int* in_sizes, int n_in,
                              void* d_out, int out_size,
                              const int* in_dtypes, int out_dtype,
                              void* const* d_out_all,
                              void* const* d_ws_all, size_t ws_bytes) {
    (void)comm; (void)in_sizes; (void)n_in; (void)out_size; (void)in_dtypes; (void)out_dtype; (void)d_out_all;
    if (nranks != NR || ws_bytes < WS_NEED) { fprintf(stderr, "unexpected configuration\n"); return; }
    static bool attr_set = false;
    constexpr int LDS_BYTES = 9 * HTB + 64;
    if (!attr_set) {
        HIP_CHECK(hipFuncSetAttribute((const void*)fused_k, hipFuncAttributeMaxDynamicSharedMemorySize, LDS_BYTES));
        attr_set = true;
    }
    Params P;
    P.x = (const signed char*)d_in[0]; P.w = (const signed char*)d_in[1];
    P.sx = (const float*)d_in[2]; P.sw = (const float*)d_in[3];
    P.out = (float*)d_out; P.rank = rank;
    for (int p = 0; p < NR; ++p) P.ws[p] = (char*)d_ws_all[p];
    fused_k<<<GRID, NTHR, LDS_BYTES, stream>>>(P);
    HIP_CHECK(hipGetLastError());
}
```

```cpp
#include <hip/hip_runtime.h>
#include <rccl/rccl.h>
#include <cstdio>

#define HIP_CHECK(expr) do { hipError_t _e = (expr); if (_e != hipSuccess) { \
    fprintf(stderr, "HIP error %s at %s:%d\n", hipGetErrorString(_e), __FILE__, __LINE__); \
    return; } } while (0)

#define LAS __attribute__((address_space(3)))
typedef int i32x4 __attribute__((ext_vector_type(4)));
typedef unsigned u32x4 __attribute__((ext_vector_type(4)));
typedef float f32x4 __attribute__((ext_vector_type(4)));

namespace {
constexpr int M = 4096, N = 16384, K = 4096, NR = 8, KS = K / NR;
constexpr int HTB = 16384;
constexpr int NKS = K / 128;
constexpr size_t HB = (size_t)NKS * HTB;
constexpr size_t SLICE = 4 * (size_t)HTB;
constexpr int GRID = 256, NPULL = 24, NTHR = 512;
constexpr int NXCH = M / 128, NWCH = N / 128;
constexpr int NCHUNK = NXCH + NWCH;
constexpr int NBLK = M / 256 + N / 256;
constexpr int NWPOS = 57;
constexpr int NPIECE = (16 + NWPOS) * 16;
constexpr int NOPIECE = 512;
constexpr unsigned READY_PER_CALL = 4 * 8;
constexpr int TILES_PER_Q = 116;
#define NSPLIT0_ 0
#define NSPLIT1_ 0
constexpr int NSPLIT0 = NSPLIT0_, NSPLIT1 = NSPLIT1_;
#define KLEAD 2u
#define NSPLIT2_ 0
constexpr int NSPLIT2 = NSPLIT2_;
constexpr int NJOBS_Q = TILES_PER_Q + NSPLIT1 + 3 * NSPLIT2;

constexpr size_t OFF_CALL = 0, OFF_TMO = 128, OFF_STAGECTR = 256, OFF_SHCTR = 384;
constexpr size_t OFF_TILECTR = 1024;
constexpr size_t OFF_PHASE = 3072;
constexpr size_t OFF_STAGED = 4096;
constexpr size_t OFF_SHDONE = 6144;
constexpr size_t OFF_READY = 8192;
constexpr size_t OFF_DBG = 65536;
constexpr size_t OFF_DATA = 1ull << 20;
constexpr size_t XF_BYTES = (size_t)M * K, WF_BYTES = (size_t)N * K;
constexpr size_t PAR_BYTES = XF_BYTES + WF_BYTES;
constexpr size_t WS_NEED = OFF_DATA + 2 * PAR_BYTES;
constexpr unsigned long long DEADLINE_TICKS = 40000000ull;

struct Params {
    const signed char* x; const signed char* w; const float* sx; const float* sw; float* out;
    char* ws[NR];
    float* outs[NR];
    int rank;
};

__device__ __forceinline__ unsigned ld_sys(const unsigned* p) { return __hip_atomic_load(p, __ATOMIC_RELAXED, __HIP_MEMORY_SCOPE_SYSTEM); }
__device__ __forceinline__ unsigned ld_agt(const unsigned* p) { return __hip_atomic_load(p, __ATOMIC_RELAXED, __HIP_MEMORY_SCOPE_AGENT); }

template <bool SYS>
__device__ __forceinline__ bool spin_ge(const unsigned* flag, unsigned want, unsigned* tmo, unsigned call) {
    const unsigned long long t0 = __builtin_amdgcn_s_memrealtime();
    while ((int)((SYS ? ld_sys(flag) : ld_agt(flag)) - want) < 0) {
        if (ld_agt(tmo) == call || __builtin_amdgcn_s_memrealtime() - t0 > DEADLINE_TICKS) {
            __hip_atomic_store(tmo, call, __ATOMIC_RELAXED, __HIP_MEMORY_SCOPE_AGENT);
            return false;
        }
        __builtin_amdgcn_s_sleep(2);
    }
    return true;
}

__device__ __forceinline__ void tr4x4(unsigned a0, unsigned a1, unsigned a2, unsigned a3, unsigned& o0, unsigned& o1, unsigned& o2, unsigned& o3) {
    const unsigned t0 = __builtin_amdgcn_perm(a1, a0, 0x05010400u);
    const unsigned t1 = __builtin_amdgcn_perm(a1, a0, 0x07030602u);
    const unsigned t2 = __builtin_amdgcn_perm(a3, a2, 0x05010400u);
    const unsigned t3 = __builtin_amdgcn_perm(a3, a2, 0x07030602u);
    o0 = __builtin_amdgcn_perm(t2, t0, 0x05040100u);
    o1 = __builtin_amdgcn_perm(t2, t0, 0x07060302u);
    o2 = __builtin_amdgcn_perm(t3, t1, 0x05040100u);
    o3 = __builtin_amdgcn_perm(t3, t1, 0x07060302u);
}

__device__ __forceinline__ void st_sc1(void* p, u32x4 v) {
    asm volatile("global_store_dwordx4 %0, %1, off sc1\n\ts_nop 1" :: "v"(p), "v"(v) : "memory");
}

#define TS() do {} while (0)
__global__ void __launch_bounds__(NTHR, 2) fused_k(Params P) {
    TS();
    extern __shared__ __attribute__((aligned(16))) char smem_[];
    LAS char* lds = (LAS char*)smem_;
    LAS unsigned* lctl = (LAS unsigned*)(lds + 9 * HTB);
    const int tid = threadIdx.x, wid = __builtin_amdgcn_readfirstlane(tid >> 6), lane = tid & 63;
    const int bid = blockIdx.x, rank = P.rank;
    char* const myws = P.ws[rank];
    unsigned* const tmo = (unsigned*)(myws + OFF_TMO);

    u32x4 sv[16];
    const bool stx = bid < NXCH, stw = bid >= NXCH && bid < NCHUNK && tid < 256;
    if (stx) {
        const signed char* src = P.x + (size_t)bid * 128 * KS;
#pragma unroll
        for (int i = 0; i < 8; ++i) {
            const int o = i * NTHR + tid;
            const int l = o & 63, kf = (o >> 6) & 1, rg = (o >> 7) & 7, ks = o >> 10;
            sv[i] = *(const u32x4*)(src + (size_t)(rg * 16 + (l & 15)) * KS + ks * 128 + kf * 64 + (l >> 4) * 16);
        }
    } else if (stw) {
        const int cg = tid & 15, kg = (tid >> 4) + ((bid - NXCH) & 1) * 16;
        const signed char* src = P.w + (size_t)(kg * 16) * N + ((bid - NXCH) >> 1) * 256 + cg * 16;
#pragma unroll
        for (int j = 0; j < 16; ++j) sv[j] = *(const u32x4*)(src + (size_t)j * N);
    }
    if (tid == 0) {
        const unsigned t = __hip_atomic_fetch_add((unsigned*)(myws + OFF_CALL), 1u, __ATOMIC_RELAXED, __HIP_MEMORY_SCOPE_AGENT);
        lctl[0] = t / GRID + 1;
    }
    __syncthreads();
    const unsigned call = __builtin_amdgcn_readfirstlane(lctl[0]);
    const unsigned par = call & 1;
    const size_t dataoff = OFF_DATA + (size_t)par * PAR_BYTES;
    char* const XF = myws + dataoff; char* const WF = XF + XF_BYTES;
    const __amdgpu_buffer_rsrc_t drs = __builtin_amdgcn_make_buffer_rsrc(XF, 0, (unsigned)PAR_BYTES, 0x00020000);

    if (bid < NCHUNK) {
        if (stx) {
            const unsigned dst = (unsigned)((size_t)bid * HB + (size_t)rank * SLICE);
#pragma unroll
            for (int i = 0; i < 8; ++i) __builtin_amdgcn_raw_buffer_store_b128(sv[i], drs, dst + (unsigned)(i * NTHR + tid) * 16u, 0, 17);
        } else {
            if (stw) {
                const int cg = tid & 15, kl = tid >> 4;
                const int dst = (kl >> 3) * HTB + (((cg >> 2) * 2) * 2 + ((kl >> 2) & 1)) * 1024 + ((kl & 3) * 16 + (cg & 3) * 4) * 16;
#pragma unroll
                for (int d = 0; d < 4; ++d) {
                    unsigned o[4][4];
#pragma unroll
                    for (int g = 0; g < 4; ++g)
                        tr4x4(sv[4 * g][d], sv[4 * g + 1][d], sv[4 * g + 2][d], sv[4 * g + 3][d], o[0][g], o[1][g], o[2][g], o[3][g]);
#pragma unroll
                    for (int i = 0; i < 4; ++i)
                        *(LAS u32x4*)(lds + dst + (i >> 1) * 2 * HTB + (i & 1) * 2048 + d * 16) = (u32x4){o[i][0], o[i][1], o[i][2], o[i][3]};
                }
            }
            __syncthreads();
            const unsigned dst = (unsigned)(XF_BYTES + (size_t)((bid - NXCH) >> 1) * 2 * HB + (size_t)rank * SLICE + (size_t)((bid - NXCH) & 1) * 2 * HTB);
#pragma unroll
            for (int i = 0; i < 8; ++i)
                __builtin_amdgcn_raw_buffer_store_b128(*(const LAS u32x4*)(lds + (i * NTHR + tid) * 16), drs, dst + (unsigned)(i >> 2) * (unsigned)HB + (unsigned)((i & 3) * NTHR + tid) * 16u, 0, 17);
        }
        asm volatile("s_waitcnt vmcnt(0)" ::: "memory");
        __syncthreads();
        if (tid == 0) {
            const unsigned t = __hip_atomic_fetch_add((unsigned*)(myws + OFF_STAGECTR), 1u, __ATOMIC_RELAXED, __HIP_MEMORY_SCOPE_AGENT);
            if (t + 1 == call * (unsigned)NCHUNK) {
#pragma unroll
                for (int q = 0; q < NR; ++q) {
                    const int p = (rank + q) & (NR - 1);
                    __hip_atomic_store((unsigned*)(P.ws[p] + OFF_STAGED + (size_t)rank * 128), call, __ATOMIC_RELAXED, __HIP_MEMORY_SCOPE_SYSTEM);
                }
            }
        }
    }
    TS();
    if (bid == GRID - 1 && tid < 8)
        __hip_atomic_store((unsigned*)(myws + OFF_TILECTR + (size_t)((par ^ 1) * 8 + tid) * 128), 0u, __ATOMIC_RELAXED, __HIP_MEMORY_SCOPE_AGENT);

    if (bid < NPULL) {
        if (wid == 0) {
            if (lane < NR) spin_ge<true>((const unsigned*)(myws + OFF_STAGED + (size_t)lane * 128), call, tmo, call);
            __builtin_amdgcn_fence(__ATOMIC_ACQUIRE, "");
            asm volatile("s_waitcnt vmcnt(0)" ::: "memory");
        }
        __syncthreads();
        const bool dead = ld_agt(tmo) == call;
        TS();
        if (!dead) {
            auto piece_off = [&](int g, unsigned& rdy) -> unsigned {
                int ks, blk; const int sub = g & 3;
                if (g < 512) { ks = g >> 7; const int b = (g >> 2) & 31; blk = (b & 1) ? 16 + (b >> 1) : (b >> 1); }
                else {
                    const int h = g - 512; int pos;
                    if (h < 512) { ks = (h >> 6) & 3; pos = 16 + (h >> 8) * 16 + ((h >> 2) & 15); }
                    else { const int r = (h - 512) >> 2; ks = r / 9; pos = 48 + r - 9 * ks; }
                    blk = 16 + (pos == 16 ? 56 + rank : pos - 1);
                }
                rdy = (unsigned)(blk * 32 + ks);
                return (unsigned)((size_t)(blk * 2 + (sub >> 1)) * HB + (size_t)ks * HTB + (size_t)(sub & 1) * 8192 + (size_t)tid * 16);
            };
            u32x4 v[7];
            unsigned rdy, off = piece_off(bid, rdy);
#pragma unroll
            for (int q = 1; q < NR; ++q) { const int p = (rank + q) & (NR - 1); v[q - 1] = *(const u32x4*)(P.ws[p] + dataoff + off + (size_t)p * SLICE); }
            for (int g = bid; g < NPIECE; g += NPULL) {
#pragma unroll
                for (int q = 1; q < NR; ++q) { const int p = (rank + q) & (NR - 1); __builtin_amdgcn_raw_buffer_store_b128(v[q - 1], drs, off + (unsigned)(p * SLICE), 0, 16); }
                const unsigned rdy_cur = rdy;
                const int gn = g + NPULL < NPIECE ? g + NPULL : g;
                off = piece_off(gn, rdy);
#pragma unroll
                for (int q = 1; q < NR; ++q) { const int p = (rank + q) & (NR - 1); v[q - 1] = *(const u32x4*)(P.ws[p] + dataoff + off + (size_t)p * SLICE); }
                asm volatile("s_waitcnt vmcnt(7)" ::: "memory");
                if (lane == 0) __hip_atomic_fetch_add((unsigned*)(myws + OFF_READY) + rdy_cur, 1u, __ATOMIC_RELAXED, __HIP_MEMORY_SCOPE_AGENT);
            }
            asm volatile("s_waitcnt vmcnt(0)" ::: "memory");
        }
        TS();
        if (wid == 0) {
            if (lane < NR && lane != rank) spin_ge<true>((const unsigned*)(myws + OFF_SHDONE + (size_t)lane * 128), call, tmo, call);
            __builtin_amdgcn_fence(__ATOMIC_ACQUIRE, "");
            asm volatile("s_waitcnt vmcnt(0)" ::: "memory");
        }
        __syncthreads();
        {
            const bool dead2 = ld_agt(tmo) == call;
            const float qn = __builtin_nanf("");
            if (dead2) {
                for (int g = bid; g < NOPIECE; g += NPULL)
                    for (int q = 1; q < NR; ++q) *(f32x4*)(P.out + (size_t)(g * 8 + wid) * N + 56 * 256 + lane * 4 + ((rank + q) & (NR - 1)) * 256) = (f32x4){qn, qn, qn, qn};
            } else {
                f32x4 v[7];
                size_t eo = (size_t)(bid * 8 + wid) * N + 56 * 256 + lane * 4;
#pragma unroll
                for (int q = 1; q < NR; ++q) { const int p = (rank + q) & (NR - 1); v[q - 1] = *(const f32x4*)(P.outs[p] + eo + p * 256); }
                for (int g = bid; g < NOPIECE; g += NPULL) {
#pragma unroll
                    for (int q = 1; q < NR; ++q) { const int p = (rank + q) & (NR - 1); *(f32x4*)(P.out + eo + p * 256) = v[q - 1]; }
                    if (g + NPULL < NOPIECE) eo += (size_t)NPULL * 8 * N;
#pragma unroll
                    for (int q = 1; q < NR; ++q) { const int p = (rank + q) & (NR - 1); v[q - 1] = *(const f32x4*)(P.outs[p] + eo + p * 256); }
                }
            }
        }
        __syncthreads();
        TS();
    }

    const int wr = wid >> 2, wc = wid & 3, fr = lane & 15, fq = lane >> 4;
    const unsigned voff = (unsigned)(wid * 1024 + lane * 16);
    const unsigned ldsw = (unsigned)wid * 1024u;
    const int aoff = wr * 8192 + lane * 16, boff = wc * 4096 + lane * 16;
    const int xcd = bid & 7;
    const float scale0 = P.sx[0] * P.sw[0];
    unsigned* const tilectr = (unsigned*)(myws + OFF_TILECTR + (size_t)(par * 8 + xcd) * 128);
    const unsigned need = call * READY_PER_CALL;
    unsigned* const phase = (unsigned*)(myws + OFF_PHASE + (size_t)xcd * 128);
    int k0 = 0;
#define SA(b, h) (((b) * 2 + (h)) * HTB)
#define SB(b, h) ((4 + (b) * 2 + (h)) * HTB)
#define STAGE(bufoff, gbase) do { _Pragma("unroll") for (int _i = 0; _i < 2; ++_i) \
        __builtin_amdgcn_global_load_lds((const unsigned*)((const char*)(gbase) + voff + _i * 8192), (LAS unsigned*)(lds + (bufoff) + ldsw + _i * 8192), 16, 0, 0); } while (0)
#define LDA(dst, b, h) do { _Pragma("unroll") for (int m = 0; m < 4; ++m) _Pragma("unroll") for (int k = 0; k < 2; ++k) dst[m][k] = *(const LAS i32x4*)(lds + SA(b, h) + aoff + m * 2048 + k * 1024); } while (0)
#define LDB(dst, b, h) do { _Pragma("unroll") for (int n = 0; n < 2; ++n) _Pragma("unroll") for (int k = 0; k < 2; ++k) dst[n][k] = *(const LAS i32x4*)(lds + SB(b, h) + boff + n * 2048 + k * 1024); } while (0)
#define MMA(ai, bj, At, Bt) do { __builtin_amdgcn_s_setprio(1); _Pragma("unroll") for (int m = 0; m < 4; ++m) _Pragma("unroll") for (int n = 0; n < 2; ++n) _Pragma("unroll") for (int k = 0; k < 2; ++k) \
        acc[ai][bj][m][n] = __builtin_amdgcn_mfma_i32_16x16x64_i8(At[m][k], Bt[n][k], acc[ai][bj][m][n], 0, 0, 0); __builtin_amdgcn_s_setprio(0); } while (0)
#define WAIT_V(n) asm volatile("s_waitcnt vmcnt(" #n ")" ::: "memory")
#define WAIT_L(n) asm volatile("s_waitcnt lgkmcnt(" #n ")" ::: "memory")
#define BAR __builtin_amdgcn_s_barrier()
#define SCHED __builtin_amdgcn_sched_barrier(0)

    constexpr unsigned NOJOB = 0xffffffffu;
    auto decode = [&](unsigned e, int qx) -> unsigned {
        e += (unsigned)(qx >> 2) * 4u;
        if (e >= (unsigned)NJOBS_Q) return NOJOB;
        unsigned u, mode;
        constexpr unsigned NF = TILES_PER_Q - NSPLIT1 - NSPLIT2;
        if (e < NF) { u = e; mode = 0; }
        else if (e < NF + 2 * NSPLIT1) { const unsigned r = e - NF; u = NF + (r >> 1); mode = 1 + (r & 1); }
        else { const unsigned r = e - NF - 2 * NSPLIT1; u = NF + NSPLIT1 + (r >> 2); mode = 4 + (r & 3); }
        const int pos = 2 * (int)(u >> 2) - (qx >> 2);
        const int wb = pos < 16 ? pos : pos == 16 ? 56 + rank : pos - 1;
        return (unsigned)((qx & 3) + 4 * (u & 3)) | ((unsigned)wb << 8) | (mode << 16);
    };
    unsigned ktab;
    {
        const int i = lane & (NKS - 1), j = i - 4, ks = (j * 37) >> 8, q = j - 7 * ks;
        ktab = (unsigned)(i < 4 ? 4 * rank + i : 4 * ((rank + 1 + q) & (NR - 1)) + ks) * (unsigned)HTB;
    }
#define KOFF(i) ((size_t)(unsigned)__builtin_amdgcn_readlane((int)ktab, (i) & (NKS - 1)))
    const unsigned* const ready = (const unsigned*)(myws + OFF_READY);

    if (tid == 0) {
        const unsigned e = __hip_atomic_fetch_add(tilectr, 1u, __ATOMIC_RELAXED, __HIP_MEMORY_SCOPE_AGENT);
        lctl[1] = decode(e, xcd);
        lctl[3] = spin_ge<true>((const unsigned*)(myws + OFF_STAGED + (size_t)rank * 128), call, tmo, call) ? 0u : 1u;
    }
    __syncthreads();
    unsigned cur = __builtin_amdgcn_readfirstlane(lctl[1]);
    bool prefetched = false, partial = false;
    int hset = 0;
    i32x4 acc[2][2][4][2];
    i32x4 At[4][2], B0[2][2], B1[2][2];

    for (;;) {
        if (cur == NOJOB) {
            __syncthreads();
            if (tid == 0) {
                unsigned c_ = NOJOB;
                for (int q = 1; q < 8 && c_ == NOJOB; ++q) {
                    unsigned* oc = (unsigned*)(myws + OFF_TILECTR + (size_t)(par * 8 + ((xcd + q) & 7)) * 128);
                    if (ld_agt(oc) < (unsigned)NJOBS_Q) c_ = decode(__hip_atomic_fetch_add(oc, 1u, __ATOMIC_RELAXED, __HIP_MEMORY_SCOPE_AGENT), (xcd + q) & 7);
                }
                lctl[1] = c_;
            }
            __syncthreads();
            cur = __builtin_amdgcn_readfirstlane(lctl[1]);
            if (cur == NOJOB) break;
        }
        const int pm = cur & 0xff, pn = (cur >> 8) & 0xff, mode = cur >> 16;
        const int jt = mode == 0 ? 0 : mode < 4 ? 1 : 2;
        const int ah = (mode >> 1) & 1, bh = mode >= 4 ? (mode & 1) : 0;
        const char* cA = XF + (size_t)(pm * 2 + ah) * HB;
        const char* cB = WF + (size_t)(pn * 2 + bh) * HB;
        if (!prefetched) {
            if (wid == 0) {
                const unsigned* f = ready + (lane < 4 ? pm * 32 + lane : (16 + pn) * 32 + (lane & 3));
                bool ok = true;
                if (lane < 8) {
                    if (mode != 0) { if (!spin_ge<false>(f, need, tmo, call)) lctl[3] = 1u; }
                    else ok = (int)(ld_agt(f) - need) >= 0;
                }
                const bool all = __all(ok);
                if (lane == 0) lctl[5] = all ? ((ld_agt(phase) + KLEAD) & (unsigned)(NKS - 2)) : 0x100u;
            }
            __syncthreads();
            const unsigned w5 = __builtin_amdgcn_readfirstlane(lctl[5]);
            partial = w5 == 0x100u; k0 = partial ? 0 : (int)w5;
            TS();
            const char* pA = cA + KOFF(k0); const char* pB = cB + KOFF(k0);
            const char* qA = cA + KOFF(k0 + 1); const char* qB = cB + KOFF(k0 + 1);
            if (mode == 0) {
                STAGE(SB(0, 0), pB); STAGE(SA(0, 0), pA); STAGE(SB(0, 1), pB + HB); STAGE(SA(0, 1), pA + HB);
                if (wr == 1) BAR;
                WAIT_V(4); BAR;
                STAGE(SB(1, 0), qB); STAGE(SA(1, 0), qA); STAGE(SB(1, 1), qB + HB);
                WAIT_V(6); BAR;
            } else if (jt == 1) {
                STAGE(0, pA); STAGE(HTB, pB); STAGE(2 * HTB, pB + HB);
                STAGE(3 * HTB, qA); STAGE(4 * HTB, qB); STAGE(5 * HTB, qB + HB);
                if (wr == 1) BAR;
                WAIT_V(8); BAR; BAR;
                hset = 0;
            } else {
                STAGE(0, pA); STAGE(HTB, pB); STAGE(2 * HTB, qA); STAGE(3 * HTB, qB);
                STAGE(4 * HTB, cA + KOFF(k0 + 2)); STAGE(5 * HTB, cB + KOFF(k0 + 2));
                if (wr == 1) BAR;
                WAIT_V(8); BAR; BAR;
                hset = 0;
            }
        }
#pragma unroll
        for (int a = 0; a < 2; ++a)
#pragma unroll
            for (int b = 0; b < 2; ++b)
#pragma unroll
                for (int m = 0; m < 4; ++m)
#pragma unroll
                    for (int n = 0; n < 2; ++n) acc[a][b][m][n] = (i32x4){0, 0, 0, 0};
        unsigned nxt = NOJOB, tk = 0;
        const char* nA = cA; const char* nB = cB;
        bool seam = false, npartial = false;
        int nk0 = k0;
#define TICKET() do { if (wid == 0 && lane == 0) { const unsigned one = 1u; asm volatile("global_atomic_add %0, %1, %2, off sc0" : "+v"(tk) : "v"(tilectr), "v"(one) : "memory"); } } while (0)
#define PROBE() do { if (wid == 0) { asm volatile("" : "+v"(tk)); const unsigned c_ = decode(__builtin_amdgcn_readfirstlane(tk), xcd); \
            if (lane == 0) lctl[4] = c_; \
            const unsigned* probe = ready + (lane < 4 ? (c_ & 15) * 32 + lane : (16 + ((c_ >> 8) & 63)) * 32 + (lane & 3)); \
            if (lane < 8) __builtin_amdgcn_global_load_lds(probe, (LAS unsigned*)(lctl + 8), 4, 0, 16); } } while (0)
#define SEAM() do { nxt = __builtin_amdgcn_readfirstlane(lctl[4]); \
            const bool all_ = __all((int)(lctl[8 + (lane & 7)] - need) >= 0); \
            const unsigned nm_ = nxt >> 16; \
            seam = nxt != NOJOB && (nm_ == 0 ? 0 : nm_ < 4 ? 1 : 2) == jt && (jt == 0 || all_); \
            if (seam) { nA = XF + (size_t)((nxt & 15) * 2 + ((nm_ >> 1) & 1)) * HB; nB = WF + (size_t)(((nxt >> 8) & 63) * 2 + (nm_ >= 4 ? (nm_ & 1) : 0)) * HB; \
                        npartial = !all_; nk0 = all_ ? k0 : 0; } } while (0)
        if (mode == 0) {
            const unsigned special = (partial ? 0x0912u : 0u) | 0xE000u;
            const char* sA = cA; const char* sB = cB; int kn = k0 + 2;
            for (int t = 0; t < NKS; t += 2) {
                if ((special >> (t >> 1)) & 1u) {
                    if (t == NKS - 2) { SEAM(); sA = nA; sB = nB; kn = nk0; }
                    else if (t == NKS - 6) TICKET();
                    else if (t == NKS - 4) PROBE();
                    else {
                        if (wid == 0 && lane < 2) {
                            const int c = t == 2 ? 0 : t == 8 ? 1 : t == 16 ? 2 : 3;
                            if (!spin_ge<false>(ready + (lane == 0 ? pm : 16 + pn) * 32 + c, need, tmo, call)) lctl[3] = 1u;
                        }
                        asm volatile("s_waitcnt lgkmcnt(0)" ::: "memory"); BAR; asm volatile("" ::: "memory");
                    }
                }
                const char* a1 = cA + KOFF(k0 + t + 1);
                const size_t o2 = KOFF(kn), o3 = KOFF(kn + 1);
                const char* a2 = sA + o2; const char* b2 = sB + o2;
                const char* a3 = sA + o3; const char* b3 = sB + o3;
                kn += 2;
                if (tid == 0) __hip_atomic_store(phase, (unsigned)((k0 + t) & (NKS - 1)), __ATOMIC_RELAXED, __HIP_MEMORY_SCOPE_AGENT);
                LDB(B0, 0, 0); SCHED; LDA(At, 0, 0); STAGE(SA(1, 1), a1 + HB);
                WAIT_L(8); WAIT_V(10); BAR; WAIT_L(0); MMA(0, 0, At, B0); BAR; SCHED;
                LDB(B1, 0, 1); STAGE(SB(0, 0), b2);
                WAIT_V(10); BAR; WAIT_L(0); MMA(0, 1, At, B1); BAR;
                LDA(At, 0, 1); STAGE(SA(0, 0), a2);
                BAR; WAIT_L(0); MMA(1, 0, At, B0); BAR; SCHED;
                STAGE(SB(0, 1), b2 + HB);
                WAIT_V(10); BAR; MMA(1, 1, At, B1); BAR;
                LDB(B0, 1, 0); SCHED; LDA(At, 1, 0); STAGE(SA(0, 1), a2 + HB);
                WAIT_L(8); WAIT_V(10); BAR; WAIT_L(0); MMA(0, 0, At, B0); BAR; SCHED;
                LDB(B1, 1, 1); STAGE(SB(1, 0), b3);
                WAIT_V(10); BAR; WAIT_L(0); MMA(0, 1, At, B1); BAR;
                LDA(At, 1, 1); STAGE(SA(1, 0), a3);
                BAR; WAIT_L(0); MMA(1, 0, At, B0); BAR; SCHED;
                STAGE(SB(1, 1), b3 + HB);
                WAIT_V(10); BAR; MMA(1, 1, At, B1); BAR;
            }
        } else if (jt == 1) {
            const char* sA = cA; const char* sB = cB;
            for (int t = 0; t < NKS; ++t) {
                if ((0x44400000u >> t) & 1u) {
                    if (t == NKS - 2) { SEAM(); sA = nA; sB = nB; }
                    else if (t == NKS - 10) TICKET();
                    else PROBE();
                }
                const size_t o2 = KOFF(k0 + t + 2);
                const char* a2 = sA + o2; const char* b2 = sB + o2;
                const int hst = hset == 0 ? 6 * HTB : hset - 3 * HTB;
#pragma unroll
                for (int n = 0; n < 2; ++n)
#pragma unroll
                    for (int k = 0; k < 2; ++k) B0[n][k] = *(const LAS i32x4*)(lds + hset + HTB + boff + n * 2048 + k * 1024);
#pragma unroll
                for (int m = 0; m < 4; ++m)
#pragma unroll
                    for (int k = 0; k < 2; ++k) At[m][k] = *(const LAS i32x4*)(lds + hset + aoff + m * 2048 + k * 1024);
                STAGE(hst, a2); STAGE(hst + HTB, b2);
                WAIT_V(10); BAR; WAIT_L(0); MMA(0, 0, At, B0); BAR; SCHED;
#pragma unroll
                for (int n = 0; n < 2; ++n)
#pragma unroll
                    for (int k = 0; k < 2; ++k) B1[n][k] = *(const LAS i32x4*)(lds + hset + 2 * HTB + boff + n * 2048 + k * 1024);
                STAGE(hst + 2 * HTB, b2 + HB);
                WAIT_V(8); BAR; WAIT_L(0); MMA(0, 1, At, B1); BAR; SCHED;
                hset = hset == 6 * HTB ? 0 : hset + 3 * HTB;
            }
        } else {
            const char* sA = cA; const char* sB = cB;
            for (int t = 0; t < NKS; ++t) {
                if ((0x20820000u >> t) & 1u) {
                    if (t == NKS - 3) { SEAM(); sA = nA; sB = nB; }
                    else if (t == NKS - 15) TICKET();
                    else PROBE();
                }
                const size_t o3 = KOFF(k0 + t + 3);
                const int hst = (hset + 6 * HTB) & (8 * HTB - 1);
#pragma unroll
                for (int n = 0; n < 2; ++n)
#pragma unroll
                    for (int k = 0; k < 2; ++k) B0[n][k] = *(const LAS i32x4*)(lds + hset + HTB + boff + n * 2048 + k * 1024);
#pragma unroll
                for (int m = 0; m < 4; ++m)
#pragma unroll
                    for (int k = 0; k < 2; ++k) At[m][k] = *(const LAS i32x4*)(lds + hset + aoff + m * 2048 + k * 1024);
                STAGE(hst, sA + o3); STAGE(hst + HTB, sB + o3);
                WAIT_L(0); WAIT_V(8); BAR; MMA(0, 0, At, B0); BAR; SCHED;
                hset = (hset + 2 * HTB) & (8 * HTB - 1);
            }
        }
        TS();
        {
            const float scale = lctl[3] != 0u ? __builtin_nanf("") : scale0;
            float* op = P.out + (size_t)(pm * 256 + ah * 128 + wr * 64 + 4 * fq) * N + pn * 256 + wc * 64 + 4 * fr;
            if (pn >= 56) {
#pragma unroll
                for (int ai = 0; ai < 2; ++ai)
#pragma unroll
                    for (int m = 0; m < 4; ++m)
#pragma unroll
                        for (int j = 0; j < 4; ++j) {
                            const f32x4 o = {(float)acc[ai][0][m][0][j] * scale, (float)acc[ai][0][m][1][j] * scale, (float)acc[ai][1][m][0][j] * scale, (float)acc[ai][1][m][1][j] * scale};
                            asm volatile("global_store_dwordx4 %0, %1, off sc0 sc1\n\ts_nop 1" :: "v"(op + (size_t)(ai * 128 + m * 16 + j) * N), "v"(o) : "memory");
                        }
                WAIT_V(0);
                if (lane == 0) {
                    const unsigned t = __hip_atomic_fetch_add((unsigned*)(myws + OFF_SHCTR), 1u, __ATOMIC_RELAXED, __HIP_MEMORY_SCOPE_AGENT);
                    if (t + 1 == call * 128u) {
#pragma unroll
                        for (int q = 1; q < NR; ++q)
                            __hip_atomic_store((unsigned*)(P.ws[(rank + q) & (NR - 1)] + OFF_SHDONE + (size_t)rank * 128), call, __ATOMIC_RELAXED, __HIP_MEMORY_SCOPE_SYSTEM);
                    }
                }
            } else if (jt != 2) {
#pragma unroll
                for (int ai = 0; ai < 2; ++ai) {
                    if (ai == 1 && jt != 0) break;
#pragma unroll
                    for (int m = 0; m < 4; ++m)
#pragma unroll
                        for (int j = 0; j < 4; ++j)
                            *(f32x4*)(op + (size_t)(ai * 128 + m * 16 + j) * N) = (f32x4){(float)acc[ai][0][m][0][j] * scale, (float)acc[ai][0][m][1][j] * scale,
                                                                                         (float)acc[ai][1][m][0][j] * scale, (float)acc[ai][1][m][1][j] * scale};
                }
            } else {
                typedef float f32x2 __attribute__((ext_vector_type(2)));
#pragma unroll
                for (int m = 0; m < 4; ++m)
#pragma unroll
                    for (int j = 0; j < 4; ++j)
                        *(f32x2*)(op + (size_t)(m * 16 + j) * N + 2 * bh) = (f32x2){(float)acc[0][0][m][0][j] * scale, (float)acc[0][0][m][1][j] * scale};
            }
        }
        TS();
        cur = nxt;
        prefetched = seam; partial = npartial; k0 = nk0;
        if (!seam) {
            WAIT_V(0);
            if (wr == 0) BAR;
            BAR;
        }
    }
}
}

extern "C" void kernel_launch(int rank, int nranks, ncclComm_t comm,
                              hipStream_t stream,
                              void* const* d_in, const int* in_sizes, int n_in,
                              void* d_out, int out_size,
                              const int* in_dtypes, int out_dtype,
                              void* const* d_out_all,
                              void* const* d_ws_all, size_t ws_bytes) {
    (void)comm; (void)in_sizes; (void)n_in; (void)out_size; (void)in_dtypes; (void)out_dtype;
    if (nranks != NR || ws_bytes < WS_NEED) { fprintf(stderr, "unexpected configuration\n"); return; }
    static bool attr_set = false;
    constexpr int LDS_BYTES = 9 * HTB + 64;
    if (!attr_set) {
        HIP_CHECK(hipFuncSetAttribute((const void*)fused_k, hipFuncAttributeMaxDynamicSharedMemorySize, LDS_BYTES));
        attr_set = true;
    }
    Params P;
    P.x = (const signed char*)d_in[0]; P.w = (const signed char*)d_in[1];
    P.sx = (const float*)d_in[2]; P.sw = (const float*)d_in[3];
    P.out = (float*)d_out; P.rank = rank;
    for (int p = 0; p < NR; ++p) { P.ws[p] = (char*)d_ws_all[p]; P.outs[p] = (float*)d_out_all[p]; }
    fused_k<<<GRID, NTHR, LDS_BYTES, stream>>>(P);
    HIP_CHECK(hipGetLastError());
}
```

```cpp
#include <hip/hip_runtime.h>
#include <rccl/rccl.h>
#include <cstdio>

#define HIP_CHECK(expr) do { hipError_t _e = (expr); if (_e != hipSuccess) { \
    fprintf(stderr, "HIP error %s at %s:%d\n", hipGetErrorString(_e), __FILE__, __LINE__); \
    return; } } while (0)

#define LAS __attribute__((address_space(3)))
typedef int i32x4 __attribute__((ext_vector_type(4)));
typedef unsigned u32x4 __attribute__((ext_vector_type(4)));
typedef float f32x4 __attribute__((ext_vector_type(4)));

namespace {
constexpr int M = 4096, N = 16384, K = 4096, NR = 8, KS = K / NR;
constexpr int HTB = 16384;
constexpr int NKS = K / 128;
constexpr size_t HB = (size_t)NKS * HTB;
constexpr size_t SLICE = 4 * (size_t)HTB;
#define NPULL_ 24
constexpr int GRID = 256, NPULL = NPULL_, NTHR = 512;
constexpr int NXCH = M / 128, NWCH = N / 128;
constexpr int NCHUNK = NXCH + NWCH;
constexpr int NWPOS = 57;
constexpr int NPIECE = (16 + NWPOS) * 16;
constexpr int NOPIECE = 512;
constexpr unsigned READY_PER_CALL = 4 * 8;
constexpr int TILES_PER_Q = 116;
#define NSPLIT1_ 0
constexpr int NSPLIT1 = NSPLIT1_;
#define STAGE_AUX 0
#define KLEAD 2u
#define NSPLIT2_ 0
constexpr int NSPLIT2 = NSPLIT2_;
constexpr int NJOBS_Q = TILES_PER_Q + NSPLIT1 + 3 * NSPLIT2;

constexpr size_t OFF_CALL = 0, OFF_TMO = 128, OFF_STAGECTR = 256, OFF_SHCTR = 384;
constexpr size_t OFF_TILECTR = 1024;
constexpr size_t OFF_PHASE = 3072;
constexpr size_t OFF_STAGED = 4096;
constexpr size_t OFF_SHDONE = 6144;
constexpr size_t OFF_READY = 8192;
constexpr size_t OFF_DBG = 65536;
constexpr size_t OFF_DATA = 1ull << 20;
constexpr size_t XF_BYTES = (size_t)M * K, WF_BYTES = (size_t)N * K;
constexpr size_t PAR_BYTES = XF_BYTES + WF_BYTES;
constexpr size_t WS_NEED = OFF_DATA + 2 * PAR_BYTES;
constexpr unsigned long long DEADLINE_TICKS = 40000000ull;

struct Params {
    const signed char* x; const signed char* w; const float* sx; const float* sw; float* out;
    char* ws[NR];
    float* outs[NR];
    int rank;
};

__device__ __forceinline__ unsigned ld_sys(const unsigned* p) { return __hip_atomic_load(p, __ATOMIC_RELAXED, __HIP_MEMORY_SCOPE_SYSTEM); }
__device__ __forceinline__ unsigned ld_agt(const unsigned* p) { return __hip_atomic_load(p, __ATOMIC_RELAXED, __HIP_MEMORY_SCOPE_AGENT); }

template <bool SYS>
__device__ __forceinline__ bool spin_ge(const unsigned* flag, unsigned want, unsigned* tmo, unsigned call) {
    const unsigned long long t0 = __builtin_amdgcn_s_memrealtime();
    while ((int)((SYS ? ld_sys(flag) : ld_agt(flag)) - want) < 0) {
        if (ld_agt(tmo) == call || __builtin_amdgcn_s_memrealtime() - t0 > DEADLINE_TICKS) {
            __hip_atomic_store(tmo, call, __ATOMIC_RELAXED, __HIP_MEMORY_SCOPE_AGENT);
            return false;
        }
        __builtin_amdgcn_s_sleep(2);
    }
    return true;
}

__device__ __forceinline__ void tr4x4(unsigned a0, unsigned a1, unsigned a2, unsigned a3, unsigned& o0, unsigned& o1, unsigned& o2, unsigned& o3) {
    const unsigned t0 = __builtin_amdgcn_perm(a1, a0, 0x05010400u);
    const unsigned t1 = __builtin_amdgcn_perm(a1, a0, 0x07030602u);
    const unsigned t2 = __builtin_amdgcn_perm(a3, a2, 0x05010400u);
    const unsigned t3 = __builtin_amdgcn_perm(a3, a2, 0x07030602u);
    o0 = __builtin_amdgcn_perm(t2, t0, 0x05040100u);
    o1 = __builtin_amdgcn_perm(t2, t0, 0x07060302u);
    o2 = __builtin_amdgcn_perm(t3, t1, 0x05040100u);
    o3 = __builtin_amdgcn_perm(t3, t1, 0x07060302u);
}

#define TS() do {} while (0)
__global__ void __launch_bounds__(NTHR, 2) fused_k(Params P) {
    TS();
    extern __shared__ __attribute__((aligned(16))) char smem_[];
    LAS char* lds = (LAS char*)smem_;
    LAS unsigned* lctl = (LAS unsigned*)(lds + 9 * HTB);
    const int tid = threadIdx.x, wid = __builtin_amdgcn_readfirstlane(tid >> 6), lane = tid & 63;
    const int bid = blockIdx.x, rank = P.rank;
    char* const myws = P.ws[rank];
    unsigned* const tmo = (unsigned*)(myws + OFF_TMO);

    u32x4 sv[16];
    const bool stx = bid < NXCH, stw = bid >= NXCH && bid < NCHUNK && tid < 256;
    if (stx) {
        const signed char* src = P.x + (size_t)bid * 128 * KS;
#pragma unroll
        for (int i = 0; i < 8; ++i) {
            const int o = i * NTHR + tid;
            const int l = o & 63, kf = (o >> 6) & 1, rg = (o >> 7) & 7, ks = o >> 10;
            sv[i] = *(const u32x4*)(src + (size_t)(rg * 16 + (l & 15)) * KS + ks * 128 + kf * 64 + (l >> 4) * 16);
        }
    } else if (stw) {
        const int cg = tid & 15, kg = (tid >> 4) + ((bid - NXCH) & 1) * 16;
        const signed char* src = P.w + (size_t)(kg * 16) * N + ((bid - NXCH) >> 1) * 256 + cg * 16;
#pragma unroll
        for (int j = 0; j < 16; ++j) sv[j] = *(const u32x4*)(src + (size_t)j * N);
    }
    if (tid == 0) {
        const unsigned t = __hip_atomic_fetch_add((unsigned*)(myws + OFF_CALL), 1u, __ATOMIC_RELAXED, __HIP_MEMORY_SCOPE_AGENT);
        lctl[0] = t / GRID + 1;
    }
    __syncthreads();
    const unsigned call = __builtin_amdgcn_readfirstlane(lctl[0]);
    const unsigned par = call & 1;
    const size_t dataoff = OFF_DATA + (size_t)par * PAR_BYTES;
    char* const XF = myws + dataoff; char* const WF = XF + XF_BYTES;
    const __amdgpu_buffer_rsrc_t drs = __builtin_amdgcn_make_buffer_rsrc(XF, 0, (unsigned)PAR_BYTES, 0x00020000);

    if (bid < NCHUNK) {
        if (stx) {
            const unsigned dst = (unsigned)((size_t)bid * HB + (size_t)rank * SLICE);
#pragma unroll
            for (int i = 0; i < 8; ++i) __builtin_amdgcn_raw_buffer_store_b128(sv[i], drs, dst + (unsigned)(i * NTHR + tid) * 16u, 0, 17);
        } else {
            if (stw) {
                const int cg = tid & 15, kl = tid >> 4;
                const int dst = (kl >> 3) * HTB + (((cg >> 2) * 2) * 2 + ((kl >> 2) & 1)) * 1024 + ((kl & 3) * 16 + (cg & 3) * 4) * 16;
#pragma unroll
                for (int d = 0; d < 4; ++d) {
                    unsigned o[4][4];
#pragma unroll
                    for (int g = 0; g < 4; ++g)
                        tr4x4(sv[4 * g][d], sv[4 * g + 1][d], sv[4 * g + 2][d], sv[4 * g + 3][d], o[0][g], o[1][g], o[2][g], o[3][g]);
#pragma unroll
                    for (int i = 0; i < 4; ++i)
                        *(LAS u32x4*)(lds + dst + (i >> 1) * 2 * HTB + (i & 1) * 2048 + d * 16) = (u32x4){o[i][0], o[i][1], o[i][2], o[i][3]};
                }
            }
            __syncthreads();
            const unsigned dst = (unsigned)(XF_BYTES + (size_t)((bid - NXCH) >> 1) * 2 * HB + (size_t)rank * SLICE + (size_t)((bid - NXCH) & 1) * 2 * HTB);
#pragma unroll
            for (int i = 0; i < 8; ++i)
                __builtin_amdgcn_raw_buffer_store_b128(*(const LAS u32x4*)(lds + (i * NTHR + tid) * 16), drs, dst + (unsigned)(i >> 2) * (unsigned)HB + (unsigned)((i & 3) * NTHR + tid) * 16u, 0, 17);
        }
        asm volatile("s_waitcnt vmcnt(0)" ::: "memory");
        __syncthreads();
        if (tid == 0) {
            const unsigned t = __hip_atomic_fetch_add((unsigned*)(myws + OFF_STAGECTR), 1u, __ATOMIC_RELAXED, __HIP_MEMORY_SCOPE_AGENT);
            if (t + 1 == call * (unsigned)NCHUNK) {
#pragma unroll
                for (int q = 0; q < NR; ++q) {
                    const int p = (rank + q) & (NR - 1);
                    __hip_atomic_store((unsigned*)(P.ws[p] + OFF_STAGED + (size_t)rank * 128), call, __ATOMIC_RELAXED, __HIP_MEMORY_SCOPE_SYSTEM);
                }
            }
        }
    }
    TS();
    if (bid == GRID - 1 && tid < 8)
        __hip_atomic_store((unsigned*)(myws + OFF_TILECTR + (size_t)((par ^ 1) * 8 + tid) * 128), 0u, __ATOMIC_RELAXED, __HIP_MEMORY_SCOPE_AGENT);

    if (bid < NPULL) {
        if (wid == 0) {
            if (lane < NR) spin_ge<true>((const unsigned*)(myws + OFF_STAGED + (size_t)lane * 128), call, tmo, call);
            __builtin_amdgcn_fence(__ATOMIC_ACQUIRE, "");
            asm volatile("s_waitcnt vmcnt(0)" ::: "memory");
        }
        __syncthreads();
        const bool dead = ld_agt(tmo) == call;
        TS();
        if (!dead) {
            auto piece_off = [&](int g, unsigned& rdy) -> unsigned {
                int ks, blk; const int sub = g & 3;
                if (g < 512) { ks = g >> 7; const int b = (g >> 2) & 31; blk = (b & 1) ? 16 + (b >> 1) : (b >> 1); }
                else {
                    const int h = g - 512; int pos;
                    if (h < 512) { ks = (h >> 6) & 3; pos = 16 + (h >> 8) * 16 + ((h >> 2) & 15); }
                    else { const int r = (h - 512) >> 2; ks = r / 9; pos = 48 + r - 9 * ks; }
                    blk = 16 + (pos == 16 ? 56 + rank : pos - 1);
                }
                rdy = (unsigned)(blk * 32 + ks);
                return (unsigned)((size_t)(blk * 2 + (sub >> 1)) * HB + (size_t)ks * HTB + (size_t)(sub & 1) * 8192 + (size_t)tid * 16);
            };
            u32x4 v[7];
            unsigned rdy, off = piece_off(bid, rdy);
#pragma unroll
            for (int q = 1; q < NR; ++q) { const int p = (rank + q) & (NR - 1); v[q - 1] = *(const u32x4*)(P.ws[p] + dataoff + off + (size_t)p * SLICE); }
            for (int g = bid; g < NPIECE; g += NPULL) {
#pragma unroll
                for (int q = 1; q < NR; ++q) { const int p = (rank + q) & (NR - 1); __builtin_amdgcn_raw_buffer_store_b128(v[q - 1], drs, off + (unsigned)(p * SLICE), 0, 16); }
                const unsigned rdy_cur = rdy;
                const int gn = g + NPULL < NPIECE ? g + NPULL : g;
                off = piece_off(gn, rdy);
#pragma unroll
                for (int q = 1; q < NR; ++q) { const int p = (rank + q) & (NR - 1); v[q - 1] = *(const u32x4*)(P.ws[p] + dataoff + off + (size_t)p * SLICE); }
                asm volatile("s_waitcnt vmcnt(7)" ::: "memory");
                if (lane == 0) __hip_atomic_fetch_add((unsigned*)(myws + OFF_READY) + rdy_cur, 1u, __ATOMIC_RELAXED, __HIP_MEMORY_SCOPE_AGENT);
            }
            asm volatile("s_waitcnt vmcnt(0)" ::: "memory");
        }
        TS();
        if (wid == 0) {
            if (lane < NR && lane != rank) spin_ge<true>((const unsigned*)(myws + OFF_SHDONE + (size_t)lane * 128), call, tmo, call);
            __builtin_amdgcn_fence(__ATOMIC_ACQUIRE, "");
            asm volatile("s_waitcnt vmcnt(0)" ::: "memory");
        }
        __syncthreads();
        {
            const bool dead2 = ld_agt(tmo) == call;
            const float qn = __builtin_nanf("");
            if (dead2) {
                for (int g = bid; g < NOPIECE; g += NPULL)
                    for (int q = 1; q < NR; ++q) *(f32x4*)(P.out + (size_t)(g * 8 + wid) * N + 56 * 256 + lane * 4 + ((rank + q) & (NR - 1)) * 256) = (f32x4){qn, qn, qn, qn};
            } else {
                f32x4 v[7];
                size_t eo = (size_t)(bid * 8 + wid) * N + 56 * 256 + lane * 4;
#pragma unroll
                for (int q = 1; q < NR; ++q) { const int p = (rank + q) & (NR - 1); v[q - 1] = *(const f32x4*)(P.outs[p] + eo + p * 256); }
                for (int g = bid; g < NOPIECE; g += NPULL) {
#pragma unroll
                    for (int q = 1; q < NR; ++q) { const int p = (rank + q) & (NR - 1); *(f32x4*)(P.out + eo + p * 256) = v[q - 1]; }
                    if (g + NPULL < NOPIECE) eo += (size_t)NPULL * 8 * N;
#pragma unroll
                    for (int q = 1; q < NR; ++q) { const int p = (rank + q) & (NR - 1); v[q - 1] = *(const f32x4*)(P.outs[p] + eo + p * 256); }
                }
            }
        }
        __syncthreads();
        TS();
    }

    const int wr = wid >> 2, wc = wid & 3, fr = lane & 15, fq = lane >> 4;
    const unsigned voff = (unsigned)(wid * 1024 + lane * 16);
    const unsigned ldsw = (unsigned)wid * 1024u;
    const int aoff = wr * 8192 + lane * 16, boff = wc * 4096 + lane * 16;
    const int xcd = bid & 7;
    const float scale0 = P.sx[0] * P.sw[0];
    unsigned* const tilectr = (unsigned*)(myws + OFF_TILECTR + (size_t)(par * 8 + xcd) * 128);
    const unsigned need = call * READY_PER_CALL;
    unsigned* const phase = (unsigned*)(myws + OFF_PHASE + (size_t)xcd * 128);
    int k0 = 0;
#define SA(b, h) (((b) * 2 + (h)) * HTB)
#define SB(b, h) ((4 + (b) * 2 + (h)) * HTB)
#define STAGE(bufoff, gbase) do { _Pragma("unroll") for (int _i = 0; _i < 2; ++_i) \
        __builtin_amdgcn_global_load_lds((const unsigned*)((const char*)(gbase) + voff + _i * 8192), (LAS unsigned*)(lds + (bufoff) + ldsw + _i * 8192), 16, 0, STAGE_AUX); } while (0)
#define LDA(dst, b, h) do { _Pragma("unroll") for (int m = 0; m < 4; ++m) _Pragma("unroll") for (int k = 0; k < 2; ++k) dst[m][k] = *(const LAS i32x4*)(lds + SA(b, h) + aoff + m * 2048 + k * 1024); } while (0)
#define LDB(dst, b, h) do { _Pragma("unroll") for (int n = 0; n < 2; ++n) _Pragma("unroll") for (int k = 0; k < 2; ++k) dst[n][k] = *(const LAS i32x4*)(lds + SB(b, h) + boff + n * 2048 + k * 1024); } while (0)
#define MMA(ai, bj, At, Bt) do { __builtin_amdgcn_s_setprio(1); _Pragma("unroll") for (int m = 0; m < 4; ++m) _Pragma("unroll") for (int n = 0; n < 2; ++n) _Pragma("unroll") for (int k = 0; k < 2; ++k) \
        acc[ai][bj][m][n] = __builtin_amdgcn_mfma_i32_16x16x64_i8(At[m][k], Bt[n][k], acc[ai][bj][m][n], 0, 0, 0); __builtin_amdgcn_s_setprio(0); } while (0)
#define WAIT_V(n) asm volatile("s_waitcnt vmcnt(" #n ")" ::: "memory")
#define WAIT_L(n) asm volatile("s_waitcnt lgkmcnt(" #n ")" ::: "memory")
#define BAR __builtin_amdgcn_s_barrier()
#define SCHED __builtin_amdgcn_sched_barrier(0)

    constexpr unsigned NOJOB = 0xffffffffu;
    auto decode = [&](unsigned e, int qx) -> unsigned {
        e += (unsigned)(qx >> 2) * 4u;
        if (e >= (unsigned)NJOBS_Q) return NOJOB;
        unsigned u, mode;
        constexpr unsigned NF = TILES_PER_Q - NSPLIT1 - NSPLIT2;
        if (e < NF) { u = e; mode = 0; }
        else if (e < NF + 2 * NSPLIT1) { const unsigned r = e - NF; u = NF + (r >> 1); mode = 1 + (r & 1); }
        else { const unsigned r = e - NF - 2 * NSPLIT1; u = NF + NSPLIT1 + (r >> 2); mode = 4 + (r & 3); }
        const int pos = 2 * (int)(u >> 2) - (qx >> 2);
        const int wb = pos < 16 ? pos : pos == 16 ? 56 + rank : pos - 1;
        return (unsigned)((qx & 3) + 4 * (u & 3)) | ((unsigned)wb << 8) | (mode << 16);
    };
    unsigned ktab;
    {
        const int i = lane & (NKS - 1), j = i - 4, ks = (j * 37) >> 8, q = j - 7 * ks;
        ktab = (unsigned)(i < 4 ? 4 * rank + i : 4 * ((rank + 1 + q) & (NR - 1)) + ks) * (unsigned)HTB;
    }
#define KOFF(i) ((size_t)(unsigned)__builtin_amdgcn_readlane((int)ktab, (i) & (NKS - 1)))
    const unsigned* const ready = (const unsigned*)(myws + OFF_READY);

    if (tid == 0) {
        const unsigned e = __hip_atomic_fetch_add(tilectr, 1u, __ATOMIC_RELAXED, __HIP_MEMORY_SCOPE_AGENT);
        lctl[1] = decode(e, xcd);
        lctl[3] = spin_ge<true>((const unsigned*)(myws + OFF_STAGED + (size_t)rank * 128), call, tmo, call) ? 0u : 1u;
    }
    __syncthreads();
    unsigned cur = __builtin_amdgcn_readfirstlane(lctl[1]);
    bool prefetched = false, partial = false;
    int hset = 0;
    i32x4 acc[2][2][4][2];
    i32x4 At[4][2], B0[2][2], B1[2][2];

    for (;;) {
        if (cur == NOJOB) {
            __syncthreads();
            if (tid == 0) {
                unsigned c_ = NOJOB;
                for (int q = 1; q < 8 && c_ == NOJOB; ++q) {
                    unsigned* oc = (unsigned*)(myws + OFF_TILECTR + (size_t)(par * 8 + ((xcd + q) & 7)) * 128);
                    if (ld_agt(oc) < (unsigned)NJOBS_Q) c_ = decode(__hip_atomic_fetch_add(oc, 1u, __ATOMIC_RELAXED, __HIP_MEMORY_SCOPE_AGENT), (xcd + q) & 7);
                }
                lctl[1] = c_;
            }
            __syncthreads();
            cur = __builtin_amdgcn_readfirstlane(lctl[1]);
            if (cur == NOJOB) break;
        }
        const int pm = cur & 0xff, pn = (cur >> 8) & 0xff, mode = (NSPLIT1 + NSPLIT2) ? (int)(cur >> 16) : 0;
        const int jt = mode == 0 ? 0 : mode < 4 ? 1 : 2;
        const int ah = (mode >> 1) & 1, bh = mode >= 4 ? (mode & 1) : 0;
        const char* cA = XF + (size_t)(pm * 2 + ah) * HB;
        const char* cB = WF + (size_t)(pn * 2 + bh) * HB;
        if (!prefetched) {
            if (wid == 0) {
                const unsigned* f = ready + (lane < 4 ? pm * 32 + lane : (16 + pn) * 32 + (lane & 3));
                bool ok = true;
                if (lane < 8) {
                    if (mode != 0) { if (!spin_ge<false>(f, need, tmo, call)) lctl[3] = 1u; }
                    else ok = (int)(ld_agt(f) - need) >= 0;
                }
                const bool all = __all(ok);
                if (lane == 0) lctl[5] = all ? ((ld_agt(phase) + KLEAD) & (unsigned)(NKS - 2)) : 0x100u;
            }
            __syncthreads();
            const unsigned w5 = __builtin_amdgcn_readfirstlane(lctl[5]);
            partial = w5 == 0x100u; k0 = partial ? 0 : (int)w5;
            TS();
            const char* pA = cA + KOFF(k0); const char* pB = cB + KOFF(k0);
            const char* qA = cA + KOFF(k0 + 1); const char* qB = cB + KOFF(k0 + 1);
            if (mode == 0) {
                STAGE(SB(0, 0), pB); STAGE(SA(0, 0), pA); STAGE(SB(0, 1), pB + HB); STAGE(SA(0, 1), pA + HB);
                if (wr == 1) BAR;
                WAIT_V(4); BAR;
                STAGE(SB(1, 0), qB); STAGE(SA(1, 0), qA); STAGE(SB(1, 1), qB + HB);
                WAIT_V(6); BAR;
            } else if (jt == 1) {
                STAGE(0, pA); STAGE(HTB, pB); STAGE(2 * HTB, pB + HB);
                STAGE(3 * HTB, qA); STAGE(4 * HTB, qB); STAGE(5 * HTB, qB + HB);
                if (wr == 1) BAR;
                WAIT_V(8); BAR; BAR;
                hset = 0;
            } else {
                STAGE(0, pA); STAGE(HTB, pB); STAGE(2 * HTB, qA); STAGE(3 * HTB, qB);
                STAGE(4 * HTB, cA + KOFF(k0 + 2)); STAGE(5 * HTB, cB + KOFF(k0 + 2));
                if (wr == 1) BAR;
                WAIT_V(8); BAR; BAR;
                hset = 0;
            }
        }
#pragma unroll
        for (int a = 0; a < 2; ++a)
#pragma unroll
            for (int b = 0; b < 2; ++b)
#pragma unroll
                for (int m = 0; m < 4; ++m)
#pragma unroll
                    for (int n = 0; n < 2; ++n) acc[a][b][m][n] = (i32x4){0, 0, 0, 0};
        unsigned nxt = NOJOB, tk = 0;
        const char* nA = cA; const char* nB = cB;
        bool seam = false, npartial = false;
        int nk0 = k0;
#define TICKET() do { if (wid == 0 && lane == 0) { const unsigned one = 1u; asm volatile("global_atomic_add %0, %1, %2, off sc0" : "+v"(tk) : "v"(tilectr), "v"(one) : "memory"); } } while (0)
#define PROBE() do { if (wid == 0) { asm volatile("" : "+v"(tk)); const unsigned c_ = decode(__builtin_amdgcn_readfirstlane(tk), xcd); \
            if (lane == 0) lctl[4] = c_; \
            const unsigned* probe = ready + (lane < 4 ? (c_ & 15) * 32 + lane : (16 + ((c_ >> 8) & 63)) * 32 + (lane & 3)); \
            if (lane < 8) __builtin_amdgcn_global_load_lds(probe, (LAS unsigned*)(lctl + 8), 4, 0, 16); } } while (0)
#define SEAM() do { nxt = __builtin_amdgcn_readfirstlane(lctl[4]); \
            const bool all_ = __all((int)(lctl[8 + (lane & 7)] - need) >= 0); \
            const unsigned nm_ = (NSPLIT1 + NSPLIT2) ? (nxt >> 16) : 0u; \
            seam = nxt != NOJOB && (nm_ == 0 ? 0 : nm_ < 4 ? 1 : 2) == jt && (jt == 0 || all_); \
            if (seam) { nA = XF + (size_t)((nxt & 15) * 2 + ((nm_ >> 1) & 1)) * HB; nB = WF + (size_t)(((nxt >> 8) & 63) * 2 + (nm_ >= 4 ? (nm_ & 1) : 0)) * HB; \
                        npartial = !all_; nk0 = all_ ? k0 : 0; } } while (0)
        if (mode == 0) {
            const unsigned special = (partial ? 0x0912u : 0u) | 0xE000u;
            const char* sA = cA; const char* sB = cB; int kn = k0 + 2;
            for (int t = 0; t < NKS; t += 2) {
                if ((special >> (t >> 1)) & 1u) {
                    if (t == NKS - 2) { SEAM(); sA = nA; sB = nB; kn = nk0; }
                    else if (t == NKS - 6) TICKET();
                    else if (t == NKS - 4) PROBE();
                    else {
                        if (wid == 0 && lane < 2) {
                            const int c = t == 2 ? 0 : t == 8 ? 1 : t == 16 ? 2 : 3;
                            if (!spin_ge<false>(ready + (lane == 0 ? pm : 16 + pn) * 32 + c, need, tmo, call)) lctl[3] = 1u;
                        }
                        asm volatile("s_waitcnt lgkmcnt(0)" ::: "memory"); BAR; asm volatile("" ::: "memory");
                        TS();
                    }
                }
                const char* a1 = cA + KOFF(k0 + t + 1);
                const size_t o2 = KOFF(kn), o3 = KOFF(kn + 1);
                const char* a2 = sA + o2; const char* b2 = sB + o2;
                const char* a3 = sA + o3; const char* b3 = sB + o3;
                kn += 2;
                if (tid == 0) __hip_atomic_store(phase, (unsigned)((k0 + t) & (NKS - 1)), __ATOMIC_RELAXED, __HIP_MEMORY_SCOPE_AGENT);
                LDB(B0, 0, 0); SCHED; LDA(At, 0, 0); STAGE(SA(1, 1), a1 + HB);
                WAIT_L(8); WAIT_V(10); BAR; WAIT_L(0); MMA(0, 0, At, B0); BAR; SCHED;
                LDB(B1, 0, 1); STAGE(SB(0, 0), b2);
                WAIT_V(10); BAR; WAIT_L(0); MMA(0, 1, At, B1); BAR;
                LDA(At, 0, 1); STAGE(SA(0, 0), a2);
                BAR; WAIT_L(0); MMA(1, 0, At, B0); BAR; SCHED;
                STAGE(SB(0, 1), b2 + HB);
                WAIT_V(10); BAR; MMA(1, 1, At, B1); BAR;
                LDB(B0, 1, 0); SCHED; LDA(At, 1, 0); STAGE(SA(0, 1), a2 + HB);
                WAIT_L(8); WAIT_V(10); BAR; WAIT_L(0); MMA(0, 0, At, B0); BAR; SCHED;
                LDB(B1, 1, 1); STAGE(SB(1, 0), b3);
                WAIT_V(10); BAR; WAIT_L(0); MMA(0, 1, At, B1); BAR;
                LDA(At, 1, 1); STAGE(SA(1, 0), a3);
                BAR; WAIT_L(0); MMA(1, 0, At, B0); BAR; SCHED;
                STAGE(SB(1, 1), b3 + HB);
                WAIT_V(10); BAR; MMA(1, 1, At, B1); BAR;
            }
        } else if (jt == 1) {
            const char* sA = cA; const char* sB = cB;
            for (int t = 0; t < NKS; ++t) {
                if ((0x44400000u >> t) & 1u) {
                    if (t == NKS - 2) { SEAM(); sA = nA; sB = nB; }
                    else if (t == NKS - 10) TICKET();
                    else PROBE();
                }
                const size_t o2 = KOFF(k0 + t + 2);
                const char* a2 = sA + o2; const char* b2 = sB + o2;
                const int hst = hset == 0 ? 6 * HTB : hset - 3 * HTB;
#pragma unroll
                for (int n = 0; n < 2; ++n)
#pragma unroll
                    for (int k = 0; k < 2; ++k) B0[n][k] = *(const LAS i32x4*)(lds + hset + HTB + boff + n * 2048 + k * 1024);
#pragma unroll
                for (int m = 0; m < 4; ++m)
#pragma unroll
                    for (int k = 0; k < 2; ++k) At[m][k] = *(const LAS i32x4*)(lds + hset + aoff + m * 2048 + k * 1024);
                STAGE(hst, a2); STAGE(hst + HTB, b2);
                WAIT_V(10); BAR; WAIT_L(0); MMA(0, 0, At, B0); BAR; SCHED;
#pragma unroll
                for (int n = 0; n < 2; ++n)
#pragma unroll
                    for (int k = 0; k < 2; ++k) B1[n][k] = *(const LAS i32x4*)(lds + hset + 2 * HTB + boff + n * 2048 + k * 1024);
                STAGE(hst + 2 * HTB, b2 + HB);
                WAIT_V(8); BAR; WAIT_L(0); MMA(0, 1, At, B1); BAR; SCHED;
                hset = hset == 6 * HTB ? 0 : hset + 3 * HTB;
            }
        } else {
            const char* sA = cA; const char* sB = cB;
            for (int t = 0; t < NKS; ++t) {
                if ((0x20820000u >> t) & 1u) {
                    if (t == NKS - 3) { SEAM(); sA = nA; sB = nB; }
                    else if (t == NKS - 15) TICKET();
                    else PROBE();
                }
                const size_t o3 = KOFF(k0 + t + 3);
                const int hst = (hset + 6 * HTB) & (8 * HTB - 1);
#pragma unroll
                for (int n = 0; n < 2; ++n)
#pragma unroll
                    for (int k = 0; k < 2; ++k) B0[n][k] = *(const LAS i32x4*)(lds + hset + HTB + boff + n * 2048 + k * 1024);
#pragma unroll
                for (int m = 0; m < 4; ++m)
#pragma unroll
                    for (int k = 0; k < 2; ++k) At[m][k] = *(const LAS i32x4*)(lds + hset + aoff + m * 2048 + k * 1024);
                STAGE(hst, sA + o3); STAGE(hst + HTB, sB + o3);
                WAIT_L(0); WAIT_V(8); BAR; MMA(0, 0, At, B0); BAR; SCHED;
                hset = (hset + 2 * HTB) & (8 * HTB - 1);
            }
        }
        TS();
        {
            const float scale = lctl[3] != 0u ? __builtin_nanf("") : scale0;
            float* op = P.out + (size_t)(pm * 256 + ah * 128 + wr * 64 + 4 * fq) * N + pn * 256 + wc * 64 + 4 * fr;
            if (pn >= 56) {
#pragma unroll
                for (int ai = 0; ai < 2; ++ai)
#pragma unroll
                    for (int m = 0; m < 4; ++m)
#pragma unroll
                        for (int j = 0; j < 4; ++j) {
                            const f32x4 o = {(float)acc[ai][0][m][0][j] * scale, (float)acc[ai][0][m][1][j] * scale, (float)acc[ai][1][m][0][j] * scale, (float)acc[ai][1][m][1][j] * scale};
                            asm volatile("global_store_dwordx4 %0, %1, off sc0 sc1\n\ts_nop 1" :: "v"(op + (size_t)(ai * 128 + m * 16 + j) * N), "v"(o) : "memory");
                        }
                WAIT_V(0);
                if (lane == 0) {
                    const unsigned t = __hip_atomic_fetch_add((unsigned*)(myws + OFF_SHCTR), 1u, __ATOMIC_RELAXED, __HIP_MEMORY_SCOPE_AGENT);
                    if (t + 1 == call * 128u) {
#pragma unroll
                        for (int q = 1; q < NR; ++q)
                            __hip_atomic_store((unsigned*)(P.ws[(rank + q) & (NR - 1)] + OFF_SHDONE + (size_t)rank * 128), call, __ATOMIC_RELAXED, __HIP_MEMORY_SCOPE_SYSTEM);
                    }
                }
            } else if (jt != 2) {
#pragma unroll
                for (int ai = 0; ai < 2; ++ai) {
                    if (ai == 1 && jt != 0) break;
#pragma unroll
                    for (int m = 0; m < 4; ++m)
#pragma unroll
                        for (int j = 0; j < 4; ++j)
                        {
                            const f32x4 o = {(float)acc[ai][0][m][0][j] * scale, (float)acc[ai][0][m][1][j] * scale, (float)acc[ai][1][m][0][j] * scale, (float)acc[ai][1][m][1][j] * scale};
                            *(f32x4*)(op + (size_t)(ai * 128 + m * 16 + j) * N) = o;
                        }
                }
            } else {
                typedef float f32x2 __attribute__((ext_vector_type(2)));
#pragma unroll
                for (int m = 0; m < 4; ++m)
#pragma unroll
                    for (int j = 0; j < 4; ++j)
                        *(f32x2*)(op + (size_t)(m * 16 + j) * N + 2 * bh) = (f32x2){(float)acc[0][0][m][0][j] * scale, (float)acc[0][0][m][1][j] * scale};
            }
        }
        TS();
        cur = nxt;
        prefetched = seam; partial = npartial; k0 = nk0;
        if (!seam) {
            WAIT_V(0);
            if (wr == 0) BAR;
            BAR;
        }
    }
}
}

extern "C" void kernel_launch(int rank, int nranks, ncclComm_t comm,
                              hipStream_t stream,
                              void* const* d_in, const int* in_sizes, int n_in,
                              void* d_out, int out_size,
                              const int* in_dtypes, int out_dtype,
                              void* const* d_out_all,
                              void* const* d_ws_all, size_t ws_bytes) {
    (void)comm; (void)in_sizes; (void)n_in; (void)out_size; (void)in_dtypes; (void)out_dtype;
    if (nranks != NR || ws_bytes < WS_NEED) { fprintf(stderr, "unexpected configuration\n"); return; }
    static bool attr_set = false;
    constexpr int LDS_BYTES = 9 * HTB + 64;
    if (!attr_set) {
        HIP_CHECK(hipFuncSetAttribute((const void*)fused_k, hipFuncAttributeMaxDynamicSharedMemorySize, LDS_BYTES));
        attr_set = true;
    }
    Params P;
    P.x = (const signed char*)d_in[0]; P.w = (const signed char*)d_in[1];
    P.sx = (const float*)d_in[2]; P.sw = (const float*)d_in[3];
    P.out = (float*)d_out; P.rank = rank;
    for (int p = 0; p < NR; ++p) { P.ws[p] = (char*)d_ws_all[p]; P.outs[p] = (float*)d_out_all[p]; }
    fused_k<<<GRID, NTHR, LDS_BYTES, stream>>>(P);
    HIP_CHECK(hipGetLastError());
}
```

```cpp
#include <hip/hip_runtime.h>
#include <rccl/rccl.h>
#include <cstdio>

#define HIP_CHECK(expr) do { hipError_t _e = (expr); if (_e != hipSuccess) { \
    fprintf(stderr, "HIP error %s at %s:%d\n", hipGetErrorString(_e), __FILE__, __LINE__); \
    return; } } while (0)

#define LAS __attribute__((address_space(3)))
typedef int i32x4 __attribute__((ext_vector_type(4)));
typedef unsigned u32x4 __attribute__((ext_vector_type(4)));
typedef float f32x4 __attribute__((ext_vector_type(4)));

namespace {
constexpr int M = 4096, N = 16384, K = 4096, NR = 8, KS = K / NR;
constexpr int HTB = 16384;
constexpr int NKS = K / 128;
constexpr size_t HB = (size_t)NKS * HTB;
constexpr size_t SLICE = 4 * (size_t)HTB;
#define NPULL_ 24
constexpr int GRID = 256, NPULL = NPULL_, NTHR = 512;
constexpr int NXCH = M / 128, NWCH = N / 128;
constexpr int NCHUNK = NXCH + NWCH;
constexpr int NWPOS = 57;
constexpr int NPIECE = (16 + NWPOS) * 16;
#define R0W_ 14
constexpr int R0W = R0W_;
constexpr int NOPIECE = 512;
constexpr unsigned READY_PER_CALL = 4 * 8;
constexpr int TILES_PER_Q = 116;
#define NSPLIT1_ 0
constexpr int NSPLIT1 = NSPLIT1_;
#define KLEAD 2u
#define NSPLIT2_ 0
constexpr int NSPLIT2 = NSPLIT2_;
constexpr int NJOBS_Q = TILES_PER_Q + NSPLIT1 + 3 * NSPLIT2;

constexpr size_t OFF_CALL = 0, OFF_TMO = 128, OFF_STAGECTR = 256, OFF_SHCTR = 384;
constexpr size_t OFF_TILECTR = 1024;
constexpr size_t OFF_PHASE = 3072;
constexpr size_t OFF_STAGED = 4096;
constexpr size_t OFF_SHDONE = 6144;
constexpr size_t OFF_READY = 8192;
constexpr size_t OFF_DBG = 65536;
constexpr size_t OFF_DATA = 1ull << 20;
constexpr size_t XF_BYTES = (size_t)M * K, WF_BYTES = (size_t)N * K;
constexpr size_t PAR_BYTES = XF_BYTES + WF_BYTES;
constexpr size_t WS_NEED = OFF_DATA + 2 * PAR_BYTES;
constexpr unsigned long long DEADLINE_TICKS = 40000000ull;

struct Params {
    const signed char* x; const signed char* w; const float* sx; const float* sw; float* out;
    char* ws[NR];
    float* outs[NR];
    int rank;
};

__device__ __forceinline__ unsigned ld_sys(const unsigned* p) { return __hip_atomic_load(p, __ATOMIC_RELAXED, __HIP_MEMORY_SCOPE_SYSTEM); }
__device__ __forceinline__ unsigned ld_agt(const unsigned* p) { return __hip_atomic_load(p, __ATOMIC_RELAXED, __HIP_MEMORY_SCOPE_AGENT); }

template <bool SYS>
__device__ __forceinline__ bool spin_ge(const unsigned* flag, unsigned want, unsigned* tmo, unsigned call) {
    const unsigned long long t0 = __builtin_amdgcn_s_memrealtime();
    while ((int)((SYS ? ld_sys(flag) : ld_agt(flag)) - want) < 0) {
        if (ld_agt(tmo) == call || __builtin_amdgcn_s_memrealtime() - t0 > DEADLINE_TICKS) {
            __hip_atomic_store(tmo, call, __ATOMIC_RELAXED, __HIP_MEMORY_SCOPE_AGENT);
            return false;
        }
        __builtin_amdgcn_s_sleep(2);
    }
    return true;
}

__device__ __forceinline__ void tr4x4(unsigned a0, unsigned a1, unsigned a2, unsigned a3, unsigned& o0, unsigned& o1, unsigned& o2, unsigned& o3) {
    const unsigned t0 = __builtin_amdgcn_perm(a1, a0, 0x05010400u);
    const unsigned t1 = __builtin_amdgcn_perm(a1, a0, 0x07030602u);
    const unsigned t2 = __builtin_amdgcn_perm(a3, a2, 0x05010400u);
    const unsigned t3 = __builtin_amdgcn_perm(a3, a2, 0x07030602u);
    o0 = __builtin_amdgcn_perm(t2, t0, 0x05040100u);
    o1 = __builtin_amdgcn_perm(t2, t0, 0x07060302u);
    o2 = __builtin_amdgcn_perm(t3, t1, 0x05040100u);
    o3 = __builtin_amdgcn_perm(t3, t1, 0x07060302u);
}

#define TS() do {} while (0)
__global__ void __launch_bounds__(NTHR, 2) fused_k(Params P) {
    TS();
    extern __shared__ __attribute__((aligned(16))) char smem_[];
    LAS char* lds = (LAS char*)smem_;
    LAS unsigned* lctl = (LAS unsigned*)(lds + 9 * HTB);
    const int tid = threadIdx.x, wid = __builtin_amdgcn_readfirstlane(tid >> 6), lane = tid & 63;
    const int bid = blockIdx.x, rank = P.rank;
    char* const myws = P.ws[rank];
    unsigned* const tmo = (unsigned*)(myws + OFF_TMO);

    u32x4 sv[16];
    const bool stx = bid < NXCH, stw = bid >= NXCH && bid < NCHUNK && tid < 256;
    if (stx) {
        const signed char* src = P.x + (size_t)bid * 128 * KS;
#pragma unroll
        for (int i = 0; i < 8; ++i) {
            const int o = i * NTHR + tid;
            const int l = o & 63, kf = (o >> 6) & 1, rg = (o >> 7) & 7, ks = o >> 10;
            sv[i] = *(const u32x4*)(src + (size_t)(rg * 16 + (l & 15)) * KS + ks * 128 + kf * 64 + (l >> 4) * 16);
        }
    } else if (stw) {
        const int cg = tid & 15, kg = (tid >> 4) + ((bid - NXCH) & 1) * 16;
        const signed char* src = P.w + (size_t)(kg * 16) * N + ((bid - NXCH) >> 1) * 256 + cg * 16;
#pragma unroll
        for (int j = 0; j < 16; ++j) sv[j] = *(const u32x4*)(src + (size_t)j * N);
    }
    if (tid == 0) {
        const unsigned t = __hip_atomic_fetch_add((unsigned*)(myws + OFF_CALL), 1u, __ATOMIC_RELAXED, __HIP_MEMORY_SCOPE_AGENT);
        lctl[0] = t / GRID + 1;
    }
    __syncthreads();
    const unsigned call = __builtin_amdgcn_readfirstlane(lctl[0]);
    const unsigned par = call & 1;
    const size_t dataoff = OFF_DATA + (size_t)par * PAR_BYTES;
    char* const XF = myws + dataoff; char* const WF = XF + XF_BYTES;
    const __amdgpu_buffer_rsrc_t drs = __builtin_amdgcn_make_buffer_rsrc(XF, 0, (unsigned)PAR_BYTES, 0x00020000);

    if (bid < NCHUNK) {
        if (stx) {
            const unsigned dst = (unsigned)((size_t)bid * HB + (size_t)rank * SLICE);
#pragma unroll
            for (int i = 0; i < 8; ++i) __builtin_amdgcn_raw_buffer_store_b128(sv[i], drs, dst + (unsigned)(i * NTHR + tid) * 16u, 0, 17);
        } else {
            if (stw) {
                const int cg = tid & 15, kl = tid >> 4;
                const int dst = (kl >> 3) * HTB + (((cg >> 2) * 2) * 2 + ((kl >> 2) & 1)) * 1024 + ((kl & 3) * 16 + (cg & 3) * 4) * 16;
#pragma unroll
                for (int d = 0; d < 4; ++d) {
                    unsigned o[4][4];
#pragma unroll
                    for (int g = 0; g < 4; ++g)
                        tr4x4(sv[4 * g][d], sv[4 * g + 1][d], sv[4 * g + 2][d], sv[4 * g + 3][d], o[0][g], o[1][g], o[2][g], o[3][g]);
#pragma unroll
                    for (int i = 0; i < 4; ++i)
                        *(LAS u32x4*)(lds + dst + (i >> 1) * 2 * HTB + (i & 1) * 2048 + d * 16) = (u32x4){o[i][0], o[i][1], o[i][2], o[i][3]};
                }
            }
            __syncthreads();
            const unsigned dst = (unsigned)(XF_BYTES + (size_t)((bid - NXCH) >> 1) * 2 * HB + (size_t)rank * SLICE + (size_t)((bid - NXCH) & 1) * 2 * HTB);
#pragma unroll
            for (int i = 0; i < 8; ++i)
                __builtin_amdgcn_raw_buffer_store_b128(*(const LAS u32x4*)(lds + (i * NTHR + tid) * 16), drs, dst + (unsigned)(i >> 2) * (unsigned)HB + (unsigned)((i & 3) * NTHR + tid) * 16u, 0, 17);
        }
        asm volatile("s_waitcnt vmcnt(0)" ::: "memory");
        __syncthreads();
        if (tid == 0) {
            const unsigned t = __hip_atomic_fetch_add((unsigned*)(myws + OFF_STAGECTR), 1u, __ATOMIC_RELAXED, __HIP_MEMORY_SCOPE_AGENT);
            if (t + 1 == call * (unsigned)NCHUNK) {
#pragma unroll
                for (int q = 0; q < NR; ++q) {
                    const int p = (rank + q) & (NR - 1);
                    __hip_atomic_store((unsigned*)(P.ws[p] + OFF_STAGED + (size_t)rank * 128), call, __ATOMIC_RELAXED, __HIP_MEMORY_SCOPE_SYSTEM);
                }
            }
        }
    }
    TS();
    if (bid == GRID - 1 && tid < 8)
        __hip_atomic_store((unsigned*)(myws + OFF_TILECTR + (size_t)((par ^ 1) * 8 + tid) * 128), 0u, __ATOMIC_RELAXED, __HIP_MEMORY_SCOPE_AGENT);

    if (bid < NPULL) {
        if (wid == 0) {
            if (lane < NR) spin_ge<true>((const unsigned*)(myws + OFF_STAGED + (size_t)lane * 128), call, tmo, call);
            __builtin_amdgcn_fence(__ATOMIC_ACQUIRE, "");
            asm volatile("s_waitcnt vmcnt(0)" ::: "memory");
        }
        __syncthreads();
        const bool dead = ld_agt(tmo) == call;
        TS();
        if (!dead) {
            auto piece_off = [&](int g, unsigned& rdy) -> unsigned {
                int ks, blk; const int sub = g & 3;
                constexpr int NB0 = 16 + R0W;
                if (g < NB0 * 16) {
                    const int r = g >> 2; ks = r / NB0; const int b = r - ks * NB0;
                    blk = b < 2 * R0W ? ((b & 1) ? 16 + (b >> 1) : (b >> 1)) : b - R0W;
                } else {
                    const int h = g - NB0 * 16; int pos;
                    constexpr int NL = NWPOS - R0W - 32;
                    if (h < 512) { ks = (h >> 6) & 3; pos = R0W + (h >> 8) * 16 + ((h >> 2) & 15); }
                    else { const int r = (h - 512) >> 2; ks = r / NL; pos = R0W + 32 + r - NL * ks; }
                    blk = 16 + (pos < 16 ? pos : pos == 16 ? 56 + rank : pos - 1);
                }
                rdy = (unsigned)(blk * 32 + ks);
                return (unsigned)((size_t)(blk * 2 + (sub >> 1)) * HB + (size_t)ks * HTB + (size_t)(sub & 1) * 8192 + (size_t)tid * 16);
            };
            u32x4 v[7];
            unsigned rdy, off = piece_off(bid, rdy);
#pragma unroll
            for (int q = 1; q < NR; ++q) { const int p = (rank + q) & (NR - 1); v[q - 1] = *(const u32x4*)(P.ws[p] + dataoff + off + (size_t)p * SLICE); }
            for (int g = bid; g < NPIECE; g += NPULL) {
#pragma unroll
                for (int q = 1; q < NR; ++q) { const int p = (rank + q) & (NR - 1); __builtin_amdgcn_raw_buffer_store_b128(v[q - 1], drs, off + (unsigned)(p * SLICE), 0, 16); }
                const unsigned rdy_cur = rdy;
                const int gn = g + NPULL < NPIECE ? g + NPULL : g;
                off = piece_off(gn, rdy);
#pragma unroll
                for (int q = 1; q < NR; ++q) { const int p = (rank + q) & (NR - 1); v[q - 1] = *(const u32x4*)(P.ws[p] + dataoff + off + (size_t)p * SLICE); }
                asm volatile("s_waitcnt vmcnt(7)" ::: "memory");
                if (lane == 0) __hip_atomic_fetch_add((unsigned*)(myws + OFF_READY) + rdy_cur, 1u, __ATOMIC_RELAXED, __HIP_MEMORY_SCOPE_AGENT);
            }
            asm volatile("s_waitcnt vmcnt(0)" ::: "memory");
        }
        TS();
        if (wid == 0) {
            if (lane < NR && lane != rank) spin_ge<true>((const unsigned*)(myws + OFF_SHDONE + (size_t)lane * 128), call, tmo, call);
            __builtin_amdgcn_fence(__ATOMIC_ACQUIRE, "");
            asm volatile("s_waitcnt vmcnt(0)" ::: "memory");
        }
        __syncthreads();
        {
            const bool dead2 = ld_agt(tmo) == call;
            const float qn = __builtin_nanf("");
            if (dead2) {
                for (int g = bid; g < NOPIECE; g += NPULL)
                    for (int q = 1; q < NR; ++q) *(f32x4*)(P.out + (size_t)(g * 8 + wid) * N + 56 * 256 + lane * 4 + ((rank + q) & (NR - 1)) * 256) = (f32x4){qn, qn, qn, qn};
            } else {
                f32x4 v[7];
                size_t eo = (size_t)(bid * 8 + wid) * N + 56 * 256 + lane * 4;
#pragma unroll
                for (int q = 1; q < NR; ++q) { const int p = (rank + q) & (NR - 1); v[q - 1] = *(const f32x4*)(P.outs[p] + eo + p * 256); }
                for (int g = bid; g < NOPIECE; g += NPULL) {
#pragma unroll
                    for (int q = 1; q < NR; ++q) { const int p = (rank + q) & (NR - 1); *(f32x4*)(P.out + eo + p * 256) = v[q - 1]; }
                    if (g + NPULL < NOPIECE) eo += (size_t)NPULL * 8 * N;
#pragma unroll
                    for (int q = 1; q < NR; ++q) { const int p = (rank + q) & (NR - 1); v[q - 1] = *(const f32x4*)(P.outs[p] + eo + p * 256); }
                }
            }
        }
        __syncthreads();
        TS();
    }

    const int wr = wid >> 2, wc = wid & 3, fr = lane & 15, fq = lane >> 4;
    const unsigned voff = (unsigned)(wid * 1024 + lane * 16);
    const unsigned ldsw = (unsigned)wid * 1024u;
    const int aoff = wr * 8192 + lane * 16, boff = wc * 4096 + lane * 16;
    const int xcd = bid & 7;
    const float scale0 = P.sx[0] * P.sw[0];
    unsigned* const tilectr = (unsigned*)(myws + OFF_TILECTR + (size_t)(par * 8 + xcd) * 128);
    const unsigned need = call * READY_PER_CALL;
    unsigned* const phase = (unsigned*)(myws + OFF_PHASE + (size_t)xcd * 128);
    int k0 = 0;
#define SA(b, h) (((b) * 2 + (h)) * HTB)
#define SB(b, h) ((4 + (b) * 2 + (h)) * HTB)
#define STAGE(bufoff, gbase) do { _Pragma("unroll") for (int _i = 0; _i < 2; ++_i) \
        __builtin_amdgcn_global_load_lds((const unsigned*)((const char*)(gbase) + voff + _i * 8192), (LAS unsigned*)(lds + (bufoff) + ldsw + _i * 8192), 16, 0, 0); } while (0)
#define LDA(dst, b, h) do { _Pragma("unroll") for (int m = 0; m < 4; ++m) _Pragma("unroll") for (int k = 0; k < 2; ++k) dst[m][k] = *(const LAS i32x4*)(lds + SA(b, h) + aoff + m * 2048 + k * 1024); } while (0)
#define LDB(dst, b, h) do { _Pragma("unroll") for (int n = 0; n < 2; ++n) _Pragma("unroll") for (int k = 0; k < 2; ++k) dst[n][k] = *(const LAS i32x4*)(lds + SB(b, h) + boff + n * 2048 + k * 1024); } while (0)
#define MMA(ai, bj, At, Bt) do { __builtin_amdgcn_s_setprio(1); _Pragma("unroll") for (int m = 0; m < 4; ++m) _Pragma("unroll") for (int n = 0; n < 2; ++n) _Pragma("unroll") for (int k = 0; k < 2; ++k) \
        acc[ai][bj][m][n] = __builtin_amdgcn_mfma_i32_16x16x64_i8(At[m][k], Bt[n][k], acc[ai][bj][m][n], 0, 0, 0); __builtin_amdgcn_s_setprio(0); } while (0)
#define WAIT_V(n) asm volatile("s_waitcnt vmcnt(" #n ")" ::: "memory")
#define WAIT_L(n) asm volatile("s_waitcnt lgkmcnt(" #n ")" ::: "memory")
#define BAR __builtin_amdgcn_s_barrier()
#define SCHED __builtin_amdgcn_sched_barrier(0)

    constexpr unsigned NOJOB = 0xffffffffu;
    auto decode = [&](unsigned e, int qx) -> unsigned {
        e += (unsigned)(qx >> 2) * 4u;
        if (e >= (unsigned)NJOBS_Q) return NOJOB;
        unsigned u, mode;
        constexpr unsigned NF = TILES_PER_Q - NSPLIT1 - NSPLIT2;
        if (e < NF) { u = e; mode = 0; }
        else if (e < NF + 2 * NSPLIT1) { const unsigned r = e - NF; u = NF + (r >> 1); mode = 1 + (r & 1); }
        else { const unsigned r = e - NF - 2 * NSPLIT1; u = NF + NSPLIT1 + (r >> 2); mode = 4 + (r & 3); }
        const int pos = 2 * (int)(u >> 2) - (qx >> 2);
        const int wb = pos < 16 ? pos : pos == 16 ? 56 + rank : pos - 1;
        return (unsigned)((qx & 3) + 4 * (u & 3)) | ((unsigned)wb << 8) | (mode << 16);
    };
    unsigned ktab;
    {
        const int i = lane & (NKS - 1), j = i - 4, ks = (j * 37) >> 8, q = j - 7 * ks;
        ktab = (unsigned)(i < 4 ? 4 * rank + i : 4 * ((rank + 1 + q) & (NR - 1)) + ks) * (unsigned)HTB;
    }
#define KOFF(i) ((size_t)(unsigned)__builtin_amdgcn_readlane((int)ktab, (i) & (NKS - 1)))
    const unsigned* const ready = (const unsigned*)(myws + OFF_READY);

    if (tid == 0) {
        const unsigned e = __hip_atomic_fetch_add(tilectr, 1u, __ATOMIC_RELAXED, __HIP_MEMORY_SCOPE_AGENT);
        lctl[1] = decode(e, xcd);
        lctl[3] = spin_ge<true>((const unsigned*)(myws + OFF_STAGED + (size_t)rank * 128), call, tmo, call) ? 0u : 1u;
    }
    __syncthreads();
    unsigned cur = __builtin_amdgcn_readfirstlane(lctl[1]);
    bool prefetched = false, partial = false;
    int hset = 0;
    i32x4 acc[2][2][4][2];
    i32x4 At[4][2], B0[2][2], B1[2][2];

    for (;;) {
        if (cur == NOJOB) {
            __syncthreads();
            if (tid == 0) {
                unsigned c_ = NOJOB;
                for (int q = 1; q < 8 && c_ == NOJOB; ++q) {
                    unsigned* oc = (unsigned*)(myws + OFF_TILECTR + (size_t)(par * 8 + ((xcd + q) & 7)) * 128);
                    if (ld_agt(oc) < (unsigned)NJOBS_Q) c_ = decode(__hip_atomic_fetch_add(oc, 1u, __ATOMIC_RELAXED, __HIP_MEMORY_SCOPE_AGENT), (xcd + q) & 7);
                }
                lctl[1] = c_;
            }
            __syncthreads();
            cur = __builtin_amdgcn_readfirstlane(lctl[1]);
            if (cur == NOJOB) break;
        }
        const int pm = cur & 0xff, pn = (cur >> 8) & 0xff, mode = (NSPLIT1 + NSPLIT2) ? (int)(cur >> 16) : 0;
        const int jt = mode == 0 ? 0 : mode < 4 ? 1 : 2;
        const int ah = (mode >> 1) & 1, bh = mode >= 4 ? (mode & 1) : 0;
        const char* cA = XF + (size_t)(pm * 2 + ah) * HB;
        const char* cB = WF + (size_t)(pn * 2 + bh) * HB;
        if (!prefetched) {
            if (wid == 0) {
                const unsigned* f = ready + (lane < 4 ? pm * 32 + lane : (16 + pn) * 32 + (lane & 3));
                bool ok = true;
                if (lane < 8) {
                    if (mode != 0) { if (!spin_ge<false>(f, need, tmo, call)) lctl[3] = 1u; }
                    else ok = (int)(ld_agt(f) - need) >= 0;
                }
                const bool all = __all(ok);
                if (lane == 0) lctl[5] = all ? ((ld_agt(phase) + KLEAD) & (unsigned)(NKS - 2)) : 0x100u;
            }
            __syncthreads();
            const unsigned w5 = __builtin_amdgcn_readfirstlane(lctl[5]);
            partial = w5 == 0x100u; k0 = partial ? 0 : (int)w5;
            TS();
            const char* pA = cA + KOFF(k0); const char* pB = cB + KOFF(k0);
            const char* qA = cA + KOFF(k0 + 1); const char* qB = cB + KOFF(k0 + 1);
            if (mode == 0) {
                STAGE(SB(0, 0), pB); STAGE(SA(0, 0), pA); STAGE(SB(0, 1), pB + HB); STAGE(SA(0, 1), pA + HB);
                if (wr == 1) BAR;
                WAIT_V(4); BAR;
                STAGE(SB(1, 0), qB); STAGE(SA(1, 0), qA); STAGE(SB(1, 1), qB + HB);
                WAIT_V(6); BAR;
            } else if (jt == 1) {
                STAGE(0, pA); STAGE(HTB, pB); STAGE(2 * HTB, pB + HB);
                STAGE(3 * HTB, qA); STAGE(4 * HTB, qB); STAGE(5 * HTB, qB + HB);
                if (wr == 1) BAR;
                WAIT_V(8); BAR; BAR;
                hset = 0;
            } else {
                STAGE(0, pA); STAGE(HTB, pB); STAGE(2 * HTB, qA); STAGE(3 * HTB, qB);
                STAGE(4 * HTB, cA + KOFF(k0 + 2)); STAGE(5 * HTB, cB + KOFF(k0 + 2));
                if (wr == 1) BAR;
                WAIT_V(8); BAR; BAR;
                hset = 0;
            }
        }
#pragma unroll
        for (int a = 0; a < 2; ++a)
#pragma unroll
            for (int b = 0; b < 2; ++b)
#pragma unroll
                for (int m = 0; m < 4; ++m)
#pragma unroll
                    for (int n = 0; n < 2; ++n) acc[a][b][m][n] = (i32x4){0, 0, 0, 0};
        unsigned nxt = NOJOB, tk = 0;
        const char* nA = cA; const char* nB = cB;
        bool seam = false, npartial = false;
        int nk0 = k0;
#define TICKET() do { if (wid == 0 && lane == 0) { const unsigned one = 1u; asm volatile("global_atomic_add %0, %1, %2, off sc0" : "+v"(tk) : "v"(tilectr), "v"(one) : "memory"); } } while (0)
#define PROBE() do { if (wid == 0) { asm volatile("" : "+v"(tk)); const unsigned c_ = decode(__builtin_amdgcn_readfirstlane(tk), xcd); \
            if (lane == 0) lctl[4] = c_; \
            const unsigned* probe = ready + (lane < 4 ? (c_ & 15) * 32 + lane : (16 + ((c_ >> 8) & 63)) * 32 + (lane & 3)); \
            if (lane < 8) __builtin_amdgcn_global_load_lds(probe, (LAS unsigned*)(lctl + 8), 4, 0, 16); } } while (0)
#define SEAM() do { nxt = __builtin_amdgcn_readfirstlane(lctl[4]); \
            const bool all_ = __all((int)(lctl[8 + (lane & 7)] - need) >= 0); \
            const unsigned nm_ = (NSPLIT1 + NSPLIT2) ? (nxt >> 16) : 0u; \
            seam = nxt != NOJOB && (nm_ == 0 ? 0 : nm_ < 4 ? 1 : 2) == jt && (jt == 0 || all_); \
            if (seam) { nA = XF + (size_t)((nxt & 15) * 2 + ((nm_ >> 1) & 1)) * HB; nB = WF + (size_t)(((nxt >> 8) & 63) * 2 + (nm_ >= 4 ? (nm_ & 1) : 0)) * HB; \
                        npartial = !all_; nk0 = all_ ? k0 : 0; } } while (0)
        if (mode == 0) {
            const unsigned special = (partial ? 0x0912u : 0u) | 0xE000u;
            const char* sA = cA; const char* sB = cB; int kn = k0 + 2;
            for (int t = 0; t < NKS; t += 2) {
                if ((special >> (t >> 1)) & 1u) {
                    if (t == NKS - 2) { SEAM(); sA = nA; sB = nB; kn = nk0; }
                    else if (t == NKS - 6) TICKET();
                    else if (t == NKS - 4) PROBE();
                    else {
                        if (wid == 0 && lane < 2) {
                            const int c = t == 2 ? 0 : t == 8 ? 1 : t == 16 ? 2 : 3;
                            if (!spin_ge<false>(ready + (lane == 0 ? pm : 16 + pn) * 32 + c, need, tmo, call)) lctl[3] = 1u;
                        }
                        asm volatile("s_waitcnt lgkmcnt(0)" ::: "memory"); BAR; asm volatile("" ::: "memory");
                        TS();
                    }
                }
                const char* a1 = cA + KOFF(k0 + t + 1);
                const size_t o2 = KOFF(kn), o3 = KOFF(kn + 1);
                const char* a2 = sA + o2; const char* b2 = sB + o2;
                const char* a3 = sA + o3; const char* b3 = sB + o3;
                kn += 2;
                if (tid == 0) __hip_atomic_store(phase, (unsigned)((k0 + t) & (NKS - 1)), __ATOMIC_RELAXED, __HIP_MEMORY_SCOPE_AGENT);
                LDB(B0, 0, 0); SCHED; LDA(At, 0, 0); STAGE(SA(1, 1), a1 + HB);
                WAIT_L(8); WAIT_V(10); BAR; WAIT_L(0); MMA(0, 0, At, B0); BAR; SCHED;
                LDB(B1, 0, 1); STAGE(SB(0, 0), b2);
                WAIT_V(10); BAR; WAIT_L(0); MMA(0, 1, At, B1); BAR;
                LDA(At, 0, 1); STAGE(SA(0, 0), a2);
                BAR; WAIT_L(0); MMA(1, 0, At, B0); BAR; SCHED;
                STAGE(SB(0, 1), b2 + HB);
                WAIT_V(10); BAR; MMA(1, 1, At, B1); BAR;
                LDB(B0, 1, 0); SCHED; LDA(At, 1, 0); STAGE(SA(0, 1), a2 + HB);
                WAIT_L(8); WAIT_V(10); BAR; WAIT_L(0); MMA(0, 0, At, B0); BAR; SCHED;
                LDB(B1, 1, 1); STAGE(SB(1, 0), b3);
                WAIT_V(10); BAR; WAIT_L(0); MMA(0, 1, At, B1); BAR;
                LDA(At, 1, 1); STAGE(SA(1, 0), a3);
                BAR; WAIT_L(0); MMA(1, 0, At, B0); BAR; SCHED;
                STAGE(SB(1, 1), b3 + HB);
                WAIT_V(10); BAR; MMA(1, 1, At, B1); BAR;
            }
        } else if (jt == 1) {
            const char* sA = cA; const char* sB = cB;
            for (int t = 0; t < NKS; ++t) {
                if ((0x44400000u >> t) & 1u) {
                    if (t == NKS - 2) { SEAM(); sA = nA; sB = nB; }
                    else if (t == NKS - 10) TICKET();
                    else PROBE();
                }
                const size_t o2 = KOFF(k0 + t + 2);
                const char* a2 = sA + o2; const char* b2 = sB + o2;
                const int hst = hset == 0 ? 6 * HTB : hset - 3 * HTB;
#pragma unroll
                for (int n = 0; n < 2; ++n)
#pragma unroll
                    for (int k = 0; k < 2; ++k) B0[n][k] = *(const LAS i32x4*)(lds + hset + HTB + boff + n * 2048 + k * 1024);
#pragma unroll
                for (int m = 0; m < 4; ++m)
#pragma unroll
                    for (int k = 0; k < 2; ++k) At[m][k] = *(const LAS i32x4*)(lds + hset + aoff + m * 2048 + k * 1024);
                STAGE(hst, a2); STAGE(hst + HTB, b2);
                WAIT_V(10); BAR; WAIT_L(0); MMA(0, 0, At, B0); BAR; SCHED;
#pragma unroll
                for (int n = 0; n < 2; ++n)
#pragma unroll
                    for (int k = 0; k < 2; ++k) B1[n][k] = *(const LAS i32x4*)(lds + hset + 2 * HTB + boff + n * 2048 + k * 1024);
                STAGE(hst + 2 * HTB, b2 + HB);
                WAIT_V(8); BAR; WAIT_L(0); MMA(0, 1, At, B1); BAR; SCHED;
                hset = hset == 6 * HTB ? 0 : hset + 3 * HTB;
            }
        } else {
            const char* sA = cA; const char* sB = cB;
            for (int t = 0; t < NKS; ++t) {
                if ((0x20820000u >> t) & 1u) {
                    if (t == NKS - 3) { SEAM(); sA = nA; sB = nB; }
                    else if (t == NKS - 15) TICKET();
                    else PROBE();
                }
                const size_t o3 = KOFF(k0 + t + 3);
                const int hst = (hset + 6 * HTB) & (8 * HTB - 1);
#pragma unroll
                for (int n = 0; n < 2; ++n)
#pragma unroll
                    for (int k = 0; k < 2; ++k) B0[n][k] = *(const LAS i32x4*)(lds + hset + HTB + boff + n * 2048 + k * 1024);
#pragma unroll
                for (int m = 0; m < 4; ++m)
#pragma unroll
                    for (int k = 0; k < 2; ++k) At[m][k] = *(const LAS i32x4*)(lds + hset + aoff + m * 2048 + k * 1024);
                STAGE(hst, sA + o3); STAGE(hst + HTB, sB + o3);
                WAIT_L(0); WAIT_V(8); BAR; MMA(0, 0, At, B0); BAR; SCHED;
                hset = (hset + 2 * HTB) & (8 * HTB - 1);
            }
        }
        TS();
        {
            const float scale = lctl[3] != 0u ? __builtin_nanf("") : scale0;
            float* op = P.out + (size_t)(pm * 256 + ah * 128 + wr * 64 + 4 * fq) * N + pn * 256 + wc * 64 + 4 * fr;
            if (pn >= 56) {
#pragma unroll
                for (int ai = 0; ai < 2; ++ai)
#pragma unroll
                    for (int m = 0; m < 4; ++m)
#pragma unroll
                        for (int j = 0; j < 4; ++j) {
                            const f32x4 o = {(float)acc[ai][0][m][0][j] * scale, (float)acc[ai][0][m][1][j] * scale, (float)acc[ai][1][m][0][j] * scale, (float)acc[ai][1][m][1][j] * scale};
                            asm volatile("global_store_dwordx4 %0, %1, off sc0 sc1\n\ts_nop 1" :: "v"(op + (size_t)(ai * 128 + m * 16 + j) * N), "v"(o) : "memory");
                        }
                WAIT_V(0);
                if (lane == 0) {
                    const unsigned t = __hip_atomic_fetch_add((unsigned*)(myws + OFF_SHCTR), 1u, __ATOMIC_RELAXED, __HIP_MEMORY_SCOPE_AGENT);
                    if (t + 1 == call * 128u) {
#pragma unroll
                        for (int q = 1; q < NR; ++q)
                            __hip_atomic_store((unsigned*)(P.ws[(rank + q) & (NR - 1)] + OFF_SHDONE + (size_t)rank * 128), call, __ATOMIC_RELAXED, __HIP_MEMORY_SCOPE_SYSTEM);
                    }
                }
            } else if (jt != 2) {
#pragma unroll
                for (int ai = 0; ai < 2; ++ai) {
                    if (ai == 1 && jt != 0) break;
#pragma unroll
                    for (int m = 0; m < 4; ++m)
#pragma unroll
                        for (int j = 0; j < 4; ++j)
                        {
                            const f32x4 o = {(float)acc[ai][0][m][0][j] * scale, (float)acc[ai][0][m][1][j] * scale, (float)acc[ai][1][m][0][j] * scale, (float)acc[ai][1][m][1][j] * scale};
                            *(f32x4*)(op + (size_t)(ai * 128 + m * 16 + j) * N) = o;
                        }
                }
            } else {
                typedef float f32x2 __attribute__((ext_vector_type(2)));
#pragma unroll
                for (int m = 0; m < 4; ++m)
#pragma unroll
                    for (int j = 0; j < 4; ++j)
                        *(f32x2*)(op + (size_t)(m * 16 + j) * N + 2 * bh) = (f32x2){(float)acc[0][0][m][0][j] * scale, (float)acc[0][0][m][1][j] * scale};
            }
        }
        TS();
        cur = nxt;
        prefetched = seam; partial = npartial; k0 = nk0;
        if (!seam) {
            WAIT_V(0);
            if (wr == 0) BAR;
            BAR;
        }
    }
}
}

extern "C" void kernel_launch(int rank, int nranks, ncclComm_t comm,
                              hipStream_t stream,
                              void* const* d_in, const int* in_sizes, int n_in,
                              void* d_out, int out_size,
                              const int* in_dtypes, int out_dtype,
                              void* const* d_out_all,
                              void* const* d_ws_all, size_t ws_bytes) {
    (void)comm; (void)in_sizes; (void)n_in; (void)out_size; (void)in_dtypes; (void)out_dtype;
    if (nranks != NR || ws_bytes < WS_NEED) { fprintf(stderr, "unexpected configuration\n"); return; }
    static bool attr_set = false;
    constexpr int LDS_BYTES = 9 * HTB + 64;
    if (!attr_set) {
        HIP_CHECK(hipFuncSetAttribute((const void*)fused_k, hipFuncAttributeMaxDynamicSharedMemorySize, LDS_BYTES));
        attr_set = true;
    }
    Params P;
    P.x = (const signed char*)d_in[0]; P.w = (const signed char*)d_in[1];
    P.sx = (const float*)d_in[2]; P.sw = (const float*)d_in[3];
    P.out = (float*)d_out; P.rank = rank;
    for (int p = 0; p < NR; ++p) { P.ws[p] = (char*)d_ws_all[p]; P.outs[p] = (float*)d_out_all[p]; }
    fused_k<<<GRID, NTHR, LDS_BYTES, stream>>>(P);
    HIP_CHECK(hipGetLastError());
}
```
